```python
import numpy as np

kernel_name = "gpu_a2a_gemm_m4096_k4096_n4096_bf16_none"


ABSMAX_RTOL = 0.008

COMPUTE_DTYPE = "bfloat16"
EPILOGUE = "none"
IN_QUANT = None
OUT_QUANT = None
IN_DTYPES = {"x": "bfloat16", "w_mat": "bfloat16"}
WORLD_SIZE = 8

def _to_bf16(x):
    u = np.ascontiguousarray(x, dtype=np.float32).view(np.uint32)
    return ((u + 0x7FFF + ((u >> 16) & 1)) & 0xFFFF0000).view(np.float32)

def setup_inputs():

    x = _to_bf16(np.random.randn(4096, 4096).astype(np.float32))
    w_mat = _to_bf16((np.random.randn(4096, 4096) / np.sqrt(4096)).astype(np.float32))
    return {"x": x, "w_mat": w_mat}

def _epilogue(y):
    return y

def reference(x, w_mat):
    y = (x @ w_mat).astype(np.float32)
    return _epilogue(y)
```

```cpp
#include <hip/hip_runtime.h>
#include <rccl/rccl.h>
#include <cstdio>

#define HIP_CHECK(expr) do { hipError_t _e = (expr); if (_e != hipSuccess) { \
    fprintf(stderr, "HIP error %s at %s:%d\n", hipGetErrorString(_e), __FILE__, __LINE__); \
    return; } } while (0)

typedef unsigned u32x4 __attribute__((ext_vector_type(4)));
typedef unsigned u32x2 __attribute__((ext_vector_type(2)));
typedef float f32x4 __attribute__((ext_vector_type(4)));
typedef __bf16 bf16x4 __attribute__((ext_vector_type(4)));
typedef __bf16 bf16x8 __attribute__((ext_vector_type(8)));
typedef __bf16 bf16x2 __attribute__((ext_vector_type(2)));
#define LAS __attribute__((address_space(3)))

namespace {
constexpr int NR = 8;
constexpr int MLOC = 512;
constexpr int KLOC = 512;
constexpr int NN = 4096;
constexpr int BM = 64, BN = 128, BK = 64;
constexpr int NSTEP = 64;
constexpr int A_TILE = BM * BK * 2;
constexpr int W_TILE = BN * BK * 2;
constexpr int SW = 6, SA = 7;
constexpr int LDS_W = 0;
constexpr int LDS_A = SW * W_TILE;
constexpr int LDS_CTL = LDS_A + SA * A_TILE;
constexpr int LDS_BYTES = LDS_CTL + 256;
constexpr size_t WS_DATA = 0;
constexpr size_t WS_BUF = (size_t)NR * MLOC * KLOC * 2;
constexpr size_t WS_FLAG = 2 * WS_BUF;
constexpr size_t WS_CNT = WS_FLAG + 64 * 128;
constexpr size_t WS_END = WS_CNT + 256 * 128;
constexpr unsigned long long DEADLINE = 50000000ull;

struct Params {
  const char* x;
  const char* w;
  char* out;
  char* ws[NR];
  int rank;
  int pad;
};

#define WAIT_V(n) asm volatile("s_waitcnt vmcnt(%0)" ::"n"(n) : "memory")
#define WAIT_L0() asm volatile("s_waitcnt lgkmcnt(0)" ::: "memory")

__device__ __forceinline__ void st16_sys(void* p, u32x4 v) {
  asm volatile("global_store_dwordx4 %0, %1, off sc0 sc1\n\ts_nop 1" ::"v"(p), "v"(v) : "memory");
}
__device__ __forceinline__ void st4_sys(void* p, unsigned v) {
  asm volatile("global_store_dword %0, %1, off sc0 sc1" ::"v"(p), "v"(v) : "memory");
}
__device__ __forceinline__ void st4_sys_l0(void* p, unsigned v) {
  asm volatile("s_mov_b64 exec, 1\n\tglobal_store_dword %0, %1, off sc0 sc1\n\ts_mov_b64 exec, -1" ::"v"(p), "v"(v) : "memory");
}
__device__ __forceinline__ unsigned ld4_sys(const void* p) {
  unsigned v;
  asm volatile("global_load_dword %0, %1, off sc0 sc1\n\ts_waitcnt vmcnt(0)" : "=&v"(v) : "v"(p) : "memory");
  return v;
}
__device__ __forceinline__ unsigned lds_ld(unsigned addr) {
  unsigned v;
  asm volatile("ds_read_b32 %0, %1\n\ts_waitcnt lgkmcnt(0)" : "=&v"(v) : "v"(addr) : "memory");
  return v;
}
__device__ __forceinline__ u32x4 lds_ld4(unsigned addr) {
  u32x4 v;
  asm volatile("ds_read_b128 %0, %1\n\ts_waitcnt lgkmcnt(0)" : "=&v"(v) : "v"(addr) : "memory");
  return v;
}
__device__ __forceinline__ void lds_st(unsigned addr, unsigned v) {
  asm volatile("ds_write_b32 %0, %1" ::"v"(addr), "v"(v) : "memory");
}
template <int SYS>
__device__ __forceinline__ void dma16(const char* sbase, unsigned voff, unsigned ldsaddr) {
  if (SYS)
    asm volatile("s_mov_b32 m0, %2\n\ts_nop 0\n\tglobal_load_lds_dwordx4 %0, %1 sc0 sc1" ::"v"(voff), "s"(sbase), "s"(ldsaddr) : "memory");
  else
    asm volatile("s_mov_b32 m0, %2\n\ts_nop 0\n\tglobal_load_lds_dwordx4 %0, %1" ::"v"(voff), "s"(sbase), "s"(ldsaddr) : "memory");
}

constexpr int C_FULLW = 0, C_FULLA = 4, C_FAIL = 8, C_DONE = 16;

__device__ __forceinline__ void step_src(int t, int rank, int rot, int& src, int& j) {
  if (t < 8) { src = rank; j = t; }
  else {
    int u = t - 8; j = u / 7; int i = u - j * 7 + rot; if (i >= 7) i -= 7;
    src = (rank + 1 + i) & 7;
  }
}

__device__ __forceinline__ int min_done(unsigned ctl) {
  u32x4 d = lds_ld4(ctl + C_DONE);
  unsigned m = min(min(d[0], d[1]), min(d[2], d[3]));
  return (int)__builtin_amdgcn_readfirstlane(m);
}

struct Frags { bf16x8 a[2][2]; bf16x4 b[4][2][2]; };

__global__ __launch_bounds__(512, 2) void a2a_gemm(Params p) {
  extern __shared__ __attribute__((aligned(1024))) char smem[];
  const unsigned lds0 = (unsigned)(size_t)(LAS char*)smem;
  const unsigned ctl = lds0 + LDS_CTL;
  const int b = blockIdx.x;
  const int wid = __builtin_amdgcn_readfirstlane(threadIdx.x >> 6);
  const int lane = threadIdx.x & 63;
  const int xcd = b & 7, idx = b >> 3;
  const int bn = xcd * 4 + (idx & 3), bm = idx >> 2;
  const int rank = p.rank;
  const int rot = (idx & 3) + ((bm & 1) ? 3 : 0);
  char* myws = p.ws[rank];

  if (threadIdx.x < 16) lds_st(ctl + threadIdx.x * 4, 0u);
  WAIT_L0();
  __builtin_amdgcn_s_barrier();

  if (wid < 4) {
    const int wm = wid >> 1, wn = wid & 1;
    const int l15 = lane & 15, g = lane >> 4, q = l15 >> 2, pp = lane & 3;
    const unsigned aoff0 = (wm * 32 + l15) * 128 + ((g ^ (l15 >> 1)) << 4);
    const unsigned boff0 = (8 * g + q) * 256 + (((wn * 8 + (pp >> 1)) ^ (q << 2) ^ ((2 * g) & 3)) << 4) + 8 * (pp & 1);
    f32x4 acc[2][4];
#pragma unroll
    for (int m = 0; m < 2; ++m)
#pragma unroll
      for (int n = 0; n < 4; ++n) acc[m][n] = (f32x4){0.f, 0.f, 0.f, 0.f};
    int have = 0;
    auto wait_full = [&](int t) {
      while (have <= t) {
        u32x2 v;
        asm volatile("ds_read_b64 %0, %1\n\ts_waitcnt lgkmcnt(0)" : "=&v"(v) : "v"(ctl) : "memory");
        have = (int)__builtin_amdgcn_readfirstlane(min(v[0], v[1]));
        if (have <= t) __builtin_amdgcn_s_sleep(1);
      }
    };
    auto load = [&](Frags& f, int sw, int sa) {
      unsigned ab = LDS_A + sa * A_TILE + aoff0;
      unsigned bb = LDS_W + sw * W_TILE + boff0;
#pragma unroll
      for (int ks = 0; ks < 2; ++ks) {
#pragma unroll
        for (int m = 0; m < 2; ++m)
          f.a[m][ks] = *(const LAS bf16x8*)((LAS char*)smem + ((ab ^ (ks << 6)) + m * 2048));
#pragma unroll
        for (int n = 0; n < 4; ++n)
#pragma unroll
          for (int t = 0; t < 2; ++t)
            f.b[n][ks][t] = __builtin_amdgcn_ds_read_tr16_b64_v4bf16((LAS bf16x4*)((LAS char*)smem + ((bb ^ ((n << 5) | (t << 4))) + t * 1024 + ks * 8192)));
      }
    };
    auto mma = [&](Frags& f) {
#pragma unroll
      for (int ks = 0; ks < 2; ++ks)
#pragma unroll
        for (int n = 0; n < 4; ++n) {
          bf16x8 wv = __builtin_shufflevector(f.b[n][ks][0], f.b[n][ks][1], 0, 1, 2, 3, 4, 5, 6, 7);
#pragma unroll
          for (int m = 0; m < 2; ++m) acc[m][n] = __builtin_amdgcn_mfma_f32_16x16x32_bf16(wv, f.a[m][ks], acc[m][n], 0, 0, 0);
        }
    };
    Frags f0, f1;
    int sw = 0, sa = 0;
    auto adv = [&]() { sw = (sw + 1 == SW) ? 0 : sw + 1; sa = (sa + 1 == SA) ? 0 : sa + 1; };
    wait_full(0);
    load(f0, sw, sa); adv();
    WAIT_L0(); __builtin_amdgcn_sched_barrier(0);
    if (lane == 0) lds_st(ctl + C_DONE + wid * 4, 1);
    for (int t = 0; t < NSTEP; t += 2) {
      wait_full(t + 1);
      load(f1, sw, sa); adv();
      mma(f0);
      __builtin_amdgcn_sched_barrier(0);
      WAIT_L0(); __builtin_amdgcn_sched_barrier(0);
      if (lane == 0) lds_st(ctl + C_DONE + wid * 4, t + 2);
      if (t + 2 < NSTEP) {
        wait_full(t + 2);
        load(f0, sw, sa); adv();
      }
      mma(f1);
      __builtin_amdgcn_sched_barrier(0);
      WAIT_L0(); __builtin_amdgcn_sched_barrier(0);
      if (lane == 0) lds_st(ctl + C_DONE + wid * 4, t + 3);
    }
    const bool fail = __builtin_amdgcn_readfirstlane(lds_ld(ctl + C_FAIL)) != 0;
    char* obase = p.out + ((size_t)(bm * BM + wm * 32 + l15) * NN + bn * BN + wn * 64) * 2;
    const unsigned coff = ((g & 1) * 16 + (g >> 1) * 8) * 2;
#pragma unroll
    for (int m = 0; m < 2; ++m)
#pragma unroll
      for (int n = 0; n < 4; n += 2) {
        f32x4 c0 = acc[m][n], c1 = acc[m][n + 1];
        unsigned a0 = __builtin_bit_cast(unsigned, (bf16x2){(__bf16)c0[0], (__bf16)c0[1]});
        unsigned a1 = __builtin_bit_cast(unsigned, (bf16x2){(__bf16)c0[2], (__bf16)c0[3]});
        unsigned b0 = __builtin_bit_cast(unsigned, (bf16x2){(__bf16)c1[0], (__bf16)c1[1]});
        unsigned b1 = __builtin_bit_cast(unsigned, (bf16x2){(__bf16)c1[2], (__bf16)c1[3]});
        auto r0 = __builtin_amdgcn_permlane16_swap(a0, b0, false, false);
        auto r1 = __builtin_amdgcn_permlane16_swap(a1, b1, false, false);
        u32x4 v = {r0[0], r1[0], r0[1], r1[1]};
        if (fail) v = (u32x4){0x7fc07fc0u, 0x7fc07fc0u, 0x7fc07fc0u, 0x7fc07fc0u};
        *(u32x4*)(obase + (size_t)m * 16 * NN * 2 + n * 32 + coff) = v;
      }
  } else if (wid == 4) {
    const int r = lane >> 4, c = lane & 15;
    unsigned voff[4];
#pragma unroll
    for (int qq = 0; qq < 4; ++qq) voff[qq] = (4 * qq + r) * (NN * 2) + ((c ^ (r << 2) ^ qq) << 4);
    const char* wb = p.w + bn * (BN * 2);
    int freed = 0, sw = 0, pub = 0;
    for (int t = 0; t < NSTEP; ++t) {
      if (t - freed >= SW) {
        freed = min_done(ctl);
        if (t - freed >= SW) {
          WAIT_V(0);
          pub = t;
          if (lane == 0) lds_st(ctl + C_FULLW, t);
          do { __builtin_amdgcn_s_sleep(1); freed = min_done(ctl); } while (t - freed >= SW);
        }
      }
      int src, j; step_src(t, rank, rot, src, j);
      const char* sb = wb + (size_t)(src * KLOC + j * BK) * (NN * 2);
      unsigned la = lds0 + LDS_W + sw * W_TILE;
#pragma unroll
      for (int i = 0; i < 16; ++i) dma16<0>(sb + (size_t)(i >> 2) * 16 * NN * 2, voff[i & 3], la + i * 1024);
      sw = (sw + 1 == SW) ? 0 : sw + 1;
      WAIT_V(32);
      if (t - 1 > pub) { pub = t - 1; if (lane == 0) lds_st(ctl + C_FULLW, t - 1); }
    }
    WAIT_V(16); if (lane == 0) lds_st(ctl + C_FULLW, NSTEP - 1);
    WAIT_V(0); if (lane == 0) lds_st(ctl + C_FULLW, NSTEP);
  } else if (wid == 5) {
    unsigned* cnt = (unsigned*)(myws + WS_CNT + b * 128);
    const int r = lane >> 3, c = lane & 7;
    unsigned voffx[2];
#pragma unroll
    for (int h = 0; h < 2; ++h) voffx[h] = (8 * h + r) * (KLOC * 2) + ((c ^ ((h << 2) | (r >> 1))) << 4);
    const char* xb = p.x + (size_t)(rank * MLOC + bm * BM) * (KLOC * 2);
    int freed = 0, sa = 0, pub = 0;
    unsigned call = 0;
    const unsigned vlin = lane * 16;
    const char* fp = myws + WS_FLAG + (bm * 8 + (lane & 7)) * 128;
    unsigned prog = 0;
    bool failed = false;
    const char* db = nullptr;
    for (int t = 0; t < NSTEP; ++t) {
      int src, j; step_src(t, rank, rot, src, j);
      bool drained = false;
      if (t - freed >= SA) {
        freed = min_done(ctl);
        if (t - freed >= SA) {
          WAIT_V(0); drained = true; pub = t;
          if (lane == 0) lds_st(ctl + C_FULLA, t);
          do { __builtin_amdgcn_s_sleep(1); freed = min_done(ctl); } while (t - freed >= SA);
        }
      }
      unsigned la = lds0 + LDS_A + sa * A_TILE;
      if (t < 8) {
        const char* sb = xb + j * (BK * 2);
#pragma unroll
        for (int i = 0; i < 8; ++i) dma16<0>(sb + (i >> 1) * 16 * KLOC * 2, voffx[i & 1], la + i * 1024);
        if (t == 1) {
          call = __builtin_amdgcn_readfirstlane(ld4_sys(cnt)) + 1;
          if (lane == 0) st4_sys(cnt, call);
          db = myws + WS_DATA + (call & 1) * WS_BUF + bm * (8 * A_TILE);
          WAIT_V(0);
        }
      } else {
        const unsigned need = call * 8 + j + 1;
        if (!failed && (unsigned)__builtin_amdgcn_readlane(prog, src) < need) {
          if (!drained) { WAIT_V(0); pub = t; if (lane == 0) lds_st(ctl + C_FULLA, t); }
          unsigned long long t0 = __builtin_amdgcn_s_memrealtime();
          while (true) {
            prog = ld4_sys(fp);
            if ((unsigned)__builtin_amdgcn_readlane(prog, src) >= need) break;
            if (__builtin_amdgcn_s_memrealtime() - t0 > DEADLINE) { failed = true; if (lane == 0) lds_st(ctl + C_FAIL, 1); break; }
            __builtin_amdgcn_s_sleep(2);
          }
        }
        const char* sb = db + (size_t)src * (MLOC * KLOC * 2) + j * A_TILE;
#pragma unroll
        for (int i = 0; i < 8; ++i) dma16<1>(sb + i * 1024, vlin, la + i * 1024);
      }
      sa = (sa + 1 == SA) ? 0 : sa + 1;
      WAIT_V(16);
      if (t - 1 > pub) { pub = t - 1; if (lane == 0) lds_st(ctl + C_FULLA, t - 1); }
    }
    WAIT_V(8); if (lane == 0) lds_st(ctl + C_FULLA, NSTEP - 1);
    WAIT_V(0); if (lane == 0) lds_st(ctl + C_FULLA, NSTEP);
  } else if (wid == 6) {
    if (b < 56) {
      unsigned* cnt = (unsigned*)(myws + WS_CNT + b * 128 + 64);
      const int dd = b % 7, rg = b / 7;
      const unsigned d = (unsigned)(rank + 1 + dd) & 7u;
      const int r = lane >> 3, c = lane & 7;
      const char* s0 = p.x + (size_t)(d * MLOC + rg * BM + r) * (KLOC * 2) + ((c ^ (r >> 1)) << 4);
      const char* s1 = p.x + (size_t)(d * MLOC + rg * BM + 8 + r) * (KLOC * 2) + ((c ^ (4 | (r >> 1))) << 4);
      u32x4 rr[3][8];
      auto L = [&](int j, u32x4* dst) {
#pragma unroll
        for (int i = 0; i < 8; ++i)
          asm volatile("global_load_dwordx4 %0, %1, off" : "=v"(dst[i]) : "v"(((i & 1) ? s1 : s0) + (i >> 1) * 16 * KLOC * 2 + j * (BK * 2)) : "memory");
      };
      unsigned cv;
      asm volatile("global_load_dword %0, %1, off sc0 sc1" : "=v"(cv) : "v"(cnt) : "memory");
      L(0, rr[0]); L(1, rr[1]); L(2, rr[2]);
      WAIT_V(16);
      asm volatile("" : "+v"(cv));
      const unsigned call = __builtin_amdgcn_readfirstlane(cv) + 1;
      char* pw = p.ws[d];
      char* dst = pw + WS_DATA + (call & 1) * WS_BUF + (size_t)rank * (MLOC * KLOC * 2) + rg * (8 * A_TILE) + lane * 16;
      char* flag = pw + WS_FLAG + (rg * 8 + rank) * 128;
      auto S = [&](int j, u32x4* src) {
#pragma unroll
        for (int i = 0; i < 8; ++i) { asm volatile("" : "+v"(src[i])); st16_sys(dst + j * A_TILE + i * 1024, src[i]); }
      };
      S(0, rr[0]);
      WAIT_V(16);
      S(1, rr[1]);
#pragma unroll
      for (int j = 2; j < 8; ++j) {
        WAIT_V(8);
        st4_sys_l0(flag, call * 8 + (j - 1));
        if (j + 1 < 8) L(j + 1, rr[(j + 1) % 3]);
        S(j, rr[j % 3]);
      }
      WAIT_V(8);
      st4_sys_l0(flag, call * 8 + 7);
      WAIT_V(1);
      st4_sys_l0(flag, call * 8 + 8); st4_sys_l0(cnt, call);
    }
  }
}
}

extern "C" void kernel_launch(int rank, int nranks, ncclComm_t comm, hipStream_t stream,
                              void* const* d_in, const int* in_sizes, int n_in,
                              void* d_out, int out_size, const int* in_dtypes, int out_dtype,
                              void* const* d_out_all, void* const* d_ws_all, size_t ws_bytes) {
  if (nranks != NR || ws_bytes < WS_END || (unsigned)rank >= (unsigned)NR) { fprintf(stderr, "bad config\n"); return; }
  static bool init = false;
  if (!init) {
    HIP_CHECK(hipFuncSetAttribute((const void*)a2a_gemm, hipFuncAttributeMaxDynamicSharedMemorySize, LDS_BYTES));
    init = true;
  }
  Params p;
  p.x = (const char*)d_in[0];
  p.w = (const char*)d_in[1];
  p.out = (char*)d_out;
  for (int i = 0; i < NR; ++i) p.ws[i] = (char*)d_ws_all[i];
  p.rank = rank;
  p.pad = 0;
  hipLaunchKernelGGL(a2a_gemm, dim3(256), dim3(512), LDS_BYTES, stream, p);
  HIP_CHECK(hipGetLastError());
}
```

```cpp
#include <hip/hip_runtime.h>
#include <rccl/rccl.h>
#include <cstdio>
#include <type_traits>

#define HIP_CHECK(expr) do { hipError_t _e = (expr); if (_e != hipSuccess) { \
    fprintf(stderr, "HIP error %s at %s:%d\n", hipGetErrorString(_e), __FILE__, __LINE__); \
    return; } } while (0)

typedef unsigned u32x4 __attribute__((ext_vector_type(4)));
typedef float f32x4 __attribute__((ext_vector_type(4)));
typedef __bf16 bf16x4 __attribute__((ext_vector_type(4)));
typedef __bf16 bf16x8 __attribute__((ext_vector_type(8)));
typedef __bf16 bf16x2 __attribute__((ext_vector_type(2)));
typedef unsigned long long u64;
#define LAS __attribute__((address_space(3)))

namespace {
constexpr int NR = 8, MLOC = 512, KLOC = 512, NN = 4096;
constexpr int W_TILE = 128 * 64 * 2;
constexpr int UNIT = 64 * 64 * 2;
constexpr int SLOT = W_TILE + 2 * UNIT;
constexpr int S = 4, D = 3;
#define FSTEPS 8
constexpr int F = FSTEPS;
constexpr int NP1 = (64 - F) / 2;
constexpr int NP = NP1 + F;
static_assert(NP1 % 4 == 0 && F % 4 == 0, "ring unroll");
constexpr int LDS_X = S * SLOT;
constexpr int LDS_CTL = LDS_X + 16384;
constexpr int LDS_CNT = LDS_CTL + 256;
constexpr int LDS_FLG = LDS_CNT + 512;
constexpr int LDS_XFL = LDS_FLG + 256;
constexpr int LDS_BYTES = LDS_XFL + 256;
#define TAU4 33
#define PUSH0 120
constexpr size_t WS_BUF = (size_t)NR * MLOC * KLOC * 2;
constexpr size_t WS_XCH = 2 * WS_BUF;
constexpr size_t WS_FLAG = WS_XCH + 256 * 16384;
constexpr size_t WS_XFLAG = WS_FLAG + 64 * 128;
constexpr size_t WS_CNT = WS_XFLAG + 256 * 128;
constexpr size_t WS_END = WS_CNT + 256 * 128;
constexpr u64 DEADLINE = 50000000ull;

struct Params {
  const char* x;
  const char* w;
  char* out;
  char* myws;
  int rank;
  int pad;
  char* ws[NR];
};

#define WAIT_V(n) asm volatile("s_waitcnt vmcnt(%0)" ::"n"(n) : "memory")
#define WAIT_L0() __builtin_amdgcn_s_waitcnt(0xc07f)
#define BAR() __builtin_amdgcn_s_barrier()
#define SB() __builtin_amdgcn_sched_barrier(0)

__device__ __forceinline__ void st16_sys(void* p, u32x4 v) {
  asm volatile("global_store_dwordx4 %0, %1, off sc0 sc1\n\ts_nop 1" ::"v"(p), "v"(v) : "memory");
}
__device__ __forceinline__ void st4_sys_l0(void* p, unsigned v) {
  asm volatile("s_mov_b64 exec, 1\n\tglobal_store_dword %0, %1, off sc0 sc1\n\ts_mov_b64 exec, -1" ::"v"(p), "v"(v) : "memory");
}
__device__ __forceinline__ unsigned ld4_sys(const void* p) {
  unsigned v;
  asm volatile("global_load_dword %0, %1, off sc0 sc1\n\ts_waitcnt vmcnt(0)" : "=&v"(v) : "v"(p) : "memory");
  return v;
}
#define GL_(o) "global_load_lds_dwordx4 %0, %1 offset:" #o " sc0 sc1\n\t"
__device__ __forceinline__ void dma8k_lin(const char* sbase, unsigned voff, unsigned ldsaddr) {
  asm volatile("s_mov_b32 m0, %2\n\ts_nop 0\n\t" GL_(-4096) GL_(-3072) GL_(-2048) GL_(-1024) GL_(0) GL_(1024) GL_(2048) GL_(3072)
               ::"v"(voff), "s"(sbase), "s"(ldsaddr) : "memory");
}
#define GV_(i, o) "global_load_lds_dwordx4 %" #i ", %8 offset:" #o "\n\t"
__device__ __forceinline__ void dma8k_v(const char* sbase, const unsigned (&v)[8], unsigned ldsaddr) {
  asm volatile("s_mov_b32 m0, %9\n\ts_nop 0\n\t" GV_(0, -4096) GV_(1, -3072) GV_(2, -2048) GV_(3, -1024) GV_(4, 0) GV_(5, 1024) GV_(6, 2048) GV_(7, 3072)
               ::"v"(v[0]), "v"(v[1]), "v"(v[2]), "v"(v[3]), "v"(v[4]), "v"(v[5]), "v"(v[6]), "v"(v[7]), "s"(sbase), "s"(ldsaddr) : "memory");
}
#define MFMA(c, x, y) asm volatile("v_mfma_f32_16x16x32_bf16 %0, %1, %2, %0" : "+v"(c) : "v"(x), "v"(y))
__device__ __forceinline__ void dma4_sys(const char* sbase, unsigned voff, unsigned ldsaddr) {
  asm volatile("s_mov_b32 m0, %2\n\ts_nop 0\n\tglobal_load_lds_dword %0, %1 sc0 sc1" ::"v"(voff), "s"(sbase), "s"(ldsaddr) : "memory");
}
__device__ __forceinline__ int first_clear(bool c) {
  const u64 m = ~__ballot(c);
  return __builtin_amdgcn_readfirstlane(m ? __builtin_ctzll(m) : 64);
}
__device__ __forceinline__ unsigned pk(float a, float b) { return __builtin_bit_cast(unsigned, (bf16x2){(__bf16)a, (__bf16)b}); }

__device__ __forceinline__ int sched_code(int e, int rank, int q, int rot) {
  if (e < 4) return rank * 8 + 2 * ((e + rot) & 3) + q;
  int u;
  if (e < NP1) {
    const int u0 = 2 * (e - 4) + q;
    const int r = u0 / 7;
    const int first = 7 * r + ((7 * r + q) & 1);
    int last = 7 * r + 6; if (last > 55 - F) last = 55 - F;
    const int c = (last - first) / 2 + 1;
    u = first + 2 * (((u0 - first) / 2 + rot) % c);
  } else {
    u = 56 - F + ((e - NP1 + rot * 2 + q) % F);
  }
  const int r = u / 7, i = u - 7 * r;
  return ((rank + 1 + i) & 7) * 8 + r;
}

#define TS(i) do {} while (0)

struct Fr { bf16x8 a[2][4]; bf16x4 b[2][4][2]; };

__global__ __launch_bounds__(448, 2) void a2a_gemm(Params p) {
  extern __shared__ __attribute__((aligned(1024))) char smem[];
  const unsigned lds0 = (unsigned)(size_t)(LAS char*)smem;
  const int b = blockIdx.x;
  const int wid = __builtin_amdgcn_readfirstlane(threadIdx.x >> 6);
  const int lane = threadIdx.x & 63;
  const int xcd = b & 7, idx = b >> 3;
  const int q = xcd >> 2, bn = (xcd & 3) * 8 + (idx & 7), bm2 = idx >> 3;
  const int rank = p.rank;
  const int rot = (bm2 + bn) & 3;
  char* myws = p.myws;
  volatile LAS unsigned* ctl = (volatile LAS unsigned*)(size_t)(lds0 + LDS_CTL);

  if (wid < 4) {
    if (wid == 0) TS(0);
    const int wm = wid >> 1, wn = wid & 1;
    const int l15 = lane & 15, g = lane >> 4, qq = l15 >> 2, pp = lane & 3;
    const unsigned aoff = W_TILE + (wm * 32 + l15) * 128 + ((g ^ (l15 >> 1)) << 4);
    const unsigned boff = (8 * g + qq) * 256 + (((wn * 8 + (pp >> 1)) ^ (qq << 2) ^ ((2 * g) & 3)) << 4) + 8 * (pp & 1);
    f32x4 acc[4][4];
#pragma unroll
    for (int m = 0; m < 4; ++m)
#pragma unroll
      for (int n = 0; n < 4; ++n) acc[m][n] = (f32x4){0.f, 0.f, 0.f, 0.f};
    unsigned ra[2][2], rb[2][8];
#pragma unroll
    for (int hh = 0; hh < 2; ++hh) {
#pragma unroll
      for (int k2 = 0; k2 < 2; ++k2) { ra[hh][k2] = lds0 + hh * 65536 + (aoff ^ (k2 << 6)); asm volatile("" : "+v"(ra[hh][k2])); }
#pragma unroll
      for (int u = 0; u < 8; ++u) { rb[hh][u] = lds0 + hh * 65536 + (boff ^ (u << 4)); asm volatile("" : "+v"(rb[hh][u])); }
    }
    auto load = [&](Fr& f, auto slc, auto mc) {
      constexpr int so = decltype(slc)::value * SLOT, hh = so / 65536, o = so % 65536, MT = decltype(mc)::value;
#pragma unroll
      for (int k2 = 0; k2 < 2; ++k2)
#pragma unroll
        for (int n = 0; n < 4; ++n) {
#pragma unroll
          for (int t = 0; t < 2; ++t)
            f.b[k2][n][t] = __builtin_amdgcn_ds_read_tr16_b64_v4bf16((LAS bf16x4*)(size_t)(rb[hh][n * 2 + t] + o + t * 1024 + k2 * 8192));
          if (n < MT) f.a[k2][n] = *(const LAS bf16x8*)(size_t)(ra[hh][k2] + o + (n >> 1) * UNIT + (n & 1) * 2048);
        }
    };
    auto period = [&](Fr& cur, Fr& nxt, auto slc, auto mcur, auto mnxt) {
      constexpr int so = decltype(slc)::value * SLOT, hh = so / 65536, o = so % 65536;
      constexpr int MC = decltype(mcur)::value, MN = decltype(mnxt)::value;
      constexpr int NM = MC * 8, NRD = MN ? 16 + MN * 2 : 0, PER = MC == 4 ? 1 : 2;
      SB();
#pragma unroll
      for (int i = 0; i < NM; ++i) {
        const int n = i / (MC * 2), m = (i >> 1) % MC, k2 = i & 1;
        bf16x8 wv = __builtin_shufflevector(cur.b[k2][n][0], cur.b[k2][n][1], 0, 1, 2, 3, 4, 5, 6, 7);
        MFMA(acc[m][n], wv, cur.a[k2][m]);
#pragma unroll
        for (int rr = i * PER; rr < i * PER + PER; ++rr) {
          if (rr < NRD) {
            if (rr >= 4 && rr < 4 + MN * 2) {
              const int mm = (rr - 4) >> 1, kk = rr & 1;
              nxt.a[kk][mm] = *(const LAS bf16x8*)(size_t)(ra[hh][kk] + o + (mm >> 1) * UNIT + (mm & 1) * 2048);
            } else {
              const int x = rr < 4 ? rr : rr - MN * 2, nn = x >> 2, kk = (x >> 1) & 1, t = x & 1;
              nxt.b[kk][nn][t] = __builtin_amdgcn_ds_read_tr16_b64_v4bf16((LAS bf16x4*)(size_t)(rb[hh][nn * 2 + t] + o + t * 1024 + kk * 8192));
            }
          }
        }
        SB();
      }
    };
    using I0 = std::integral_constant<int, 0>; using I1 = std::integral_constant<int, 1>;
    using I2 = std::integral_constant<int, 2>; using I3 = std::integral_constant<int, 3>; using I4 = std::integral_constant<int, 4>;
#define ENDP() do { WAIT_L0(); BAR(); } while (0)
    Fr f0, f1;
    BAR();
    if (wid == 0) TS(3);
    load(f0, I0{}, I4{});
    const unsigned call = __builtin_amdgcn_readfirstlane(ctl[64]) + 1;
    ENDP();
    for (int pr = 0; pr < NP1 - 4; pr += 4) {
      period(f0, f1, I1{}, I4{}, I4{}); ENDP();
      period(f1, f0, I2{}, I4{}, I4{}); ENDP();
      period(f0, f1, I3{}, I4{}, I4{}); ENDP();
      period(f1, f0, I0{}, I4{}, I4{}); ENDP();
    }
    period(f0, f1, I1{}, I4{}, I4{}); ENDP();
    period(f1, f0, I2{}, I4{}, I4{}); ENDP();
    period(f0, f1, I3{}, I4{}, I4{}); ENDP();
    period(f1, f0, I0{}, I4{}, I2{}); ENDP();
    asm volatile("s_nop 7" : "+v"(acc[3][3]), "+v"(acc[2][3]));
    if (wid == 0) TS(6);
    {
      char* xd = myws + WS_XCH + (size_t)(b ^ 4) * 16384 + wid * 4096 + lane * 16;
#pragma unroll
      for (int m = 0; m < 2; ++m)
#pragma unroll
        for (int n = 0; n < 4; n += 2) {
          const f32x4 c0 = acc[2 + m][n], c1 = acc[2 + m][n + 1];
          st16_sys(xd + (m * 2 + (n >> 1)) * 1024, (u32x4){pk(c0[0], c0[1]), pk(c0[2], c0[3]), pk(c1[0], c1[1]), pk(c1[2], c1[3])});
        }
    }
    period(f0, f1, I1{}, I2{}, I2{}); ENDP();
    period(f1, f0, I2{}, I2{}, I2{});
    WAIT_V(0);
    st4_sys_l0(myws + WS_XFLAG + (b ^ 4) * 128 + wid * 4, call);
    ENDP();
    period(f0, f1, I3{}, I2{}, I2{}); ENDP();
    for (int pr = 4; pr < F; pr += 4) {
      period(f1, f0, I0{}, I2{}, I2{}); ENDP();
      period(f0, f1, I1{}, I2{}, I2{}); ENDP();
      period(f1, f0, I2{}, I2{}, I2{}); ENDP();
      period(f0, f1, I3{}, I2{}, I2{}); ENDP();
    }
    u32x4 xp[4];
#pragma unroll
    for (int i = 0; i < 4; ++i) xp[i] = *(const LAS u32x4*)(size_t)(lds0 + LDS_X + wid * 4096 + i * 1024 + lane * 16);
    const bool fail = __builtin_amdgcn_readfirstlane(ctl[0]) != 0;
    period(f1, f0, I0{}, I2{}, I0{});
    asm volatile("s_nop 7\n\ts_nop 1" : "+v"(acc[1][3]), "+v"(acc[0][3]), "+v"(acc[1][2]));
    if (wid == 0) TS(1);
    char* obase = p.out + ((size_t)((bm2 * 2 + q) * 64 + wm * 32 + l15) * NN + bn * 128 + wn * 64) * 2 + ((g & 1) * 16 + (g >> 1) * 8) * 2;
#pragma unroll
    for (int m = 0; m < 2; ++m)
#pragma unroll
      for (int n = 0; n < 4; n += 2) {
        const f32x4 c0 = acc[m][n], c1 = acc[m][n + 1];
        const u32x4 x = xp[m * 2 + (n >> 1)];
        auto lo = [](unsigned v) { return __uint_as_float(v << 16); };
        auto hi = [](unsigned v) { return __uint_as_float(v & 0xffff0000u); };
        const unsigned a0 = pk(c0[0] + lo(x[0]), c0[1] + hi(x[0])), a1 = pk(c0[2] + lo(x[1]), c0[3] + hi(x[1]));
        const unsigned b0 = pk(c1[0] + lo(x[2]), c1[1] + hi(x[2])), b1 = pk(c1[2] + lo(x[3]), c1[3] + hi(x[3]));
        auto r0 = __builtin_amdgcn_permlane16_swap(a0, b0, false, false);
        auto r1 = __builtin_amdgcn_permlane16_swap(a1, b1, false, false);
        u32x4 v = {r0[0], r1[0], r0[1], r1[1]};
        if (fail) v = (u32x4){0x7fc07fc0u, 0x7fc07fc0u, 0x7fc07fc0u, 0x7fc07fc0u};
        *(u32x4*)(obase + (size_t)m * 16 * NN * 2 + n * 32) = v;
      }
  } else if (wid == 4) {
    const int r = lane >> 4, c = lane & 15;
    unsigned voff[8];
#pragma unroll
    for (int u = 0; u < 8; ++u) voff[u] = (4 * u + r) * (NN * 2) + ((c ^ (r << 2) ^ (u & 3)) << 4) + 4096 - u * 1024;
    const char* base = p.w + bn * 256;
    auto issue = [&](const char* sb, unsigned la) { dma8k_v(sb, voff, la); dma8k_v(sb + 32 * NN * 2, voff, la + 8192); };
    auto src = [&](int code) { return base + (size_t)code * (64 * NN * 2); };
    if (b < 32) __builtin_amdgcn_s_sleep(5);
    issue(src(rank * 8 + 2 * (rot & 3) + q), lds0 + 4096);
    issue(src(rank * 8 + 2 * ((1 + rot) & 3) + q), lds0 + 4096 + SLOT);
    const int tab = sched_code(lane < NP ? lane : 0, rank, q, rot);
    WAIT_V(16);
    BAR();
    issue(src(rank * 8 + 2 * ((2 + rot) & 3) + q), lds0 + 4096 + 2 * SLOT);
    WAIT_V(16);
    BAR();
    issue(src(rank * 8 + 2 * ((3 + rot) & 3) + q), lds0 + 4096 + 3 * SLOT);
    const char* sb = src(__builtin_amdgcn_readlane(tab, 4));
    unsigned la = lds0 + 4096;
    for (int k = 1; k < NP - D; ++k) {
      issue(sb, la);
      sb = src(__builtin_amdgcn_readlane(tab, k + D + 1));
      la = lds0 + 4096 + ((k + D + 1) & 3) * SLOT;
      asm volatile("" : "+s"(sb), "+s"(la));
      WAIT_V(32);
      BAR();
    }
    WAIT_V(16); BAR();
    WAIT_V(0); BAR();
  } else {
    const int r = lane >> 3, c = lane & 7;
    unsigned voffx[8];
#pragma unroll
    for (int u = 0; u < 8; ++u) voffx[u] = (8 * u + r) * (KLOC * 2) + ((c ^ (((u & 1) << 2) | (r >> 1))) << 4) + 4096 - u * 1024;
    char* cnt = myws + WS_CNT + b * 128;
    volatile LAS unsigned* fl = (volatile LAS unsigned*)(size_t)(lds0 + LDS_FLG + lane * 4);
    if (wid == 5) {
      const char* xb = p.x + (size_t)(rank * MLOC + bm2 * 128) * (KLOC * 2);
      auto issue_local = [&](int e) {
        const char* sb = xb + (2 * ((e + rot) & 3) + q) * 128;
        const unsigned la = lds0 + e * SLOT + W_TILE + 4096;
        dma8k_v(sb + q * (64 * KLOC * 2), voffx, la);
        dma8k_v(sb + (q ^ 1) * (64 * KLOC * 2), voffx, la + UNIT);
      };
      fl[0] = 0; fl[64] = 0; ctl[0] = 0;
      if (b < 32) __builtin_amdgcn_s_sleep(5);
      dma4_sys(cnt, 0u, lds0 + LDS_CNT);
      issue_local(0); issue_local(1);
      const int tab = sched_code(lane < NP ? lane : 0, rank, q, rot);
      WAIT_V(16);
      const unsigned call = __builtin_amdgcn_readfirstlane(ctl[64]) + 1;
      const char* db = myws + (call & 1) * WS_BUF + (size_t)(bm2 * 2) * 8 * UNIT + 4096;
      const char* fp = myws + WS_FLAG + bm2 * 1024;
      const unsigned fvo = (lane >> 3) * 128 + (lane & 7) * 4;
      const unsigned vlin = lane * 16;
      volatile LAS unsigned* fe = (volatile LAS unsigned*)(size_t)(lds0 + LDS_FLG + tab * 4);
      const bool nocheck = lane < 4 || lane >= NP;
      const unsigned aoffs = (tab >> 3) * (MLOC * KLOC * 2) + (tab & 7) * UNIT;
      int avail = 4;
      auto poll_issue = [&]() { dma4_sys(fp, fvo, lds0 + LDS_FLG); };
      auto poll_read = [&]() { avail = first_clear(nocheck || *fe >= call); };
      WAIT_L0();
      BAR();
      issue_local(2);
      WAIT_V(16);
      BAR();
      issue_local(3);
      poll_issue();
      WAIT_V(17);
      BAR();
      st4_sys_l0(cnt, call);
      int na = 4;
      int xst = 0;
      auto issue = [&]() {
        const char* sb = db + (unsigned)__builtin_amdgcn_readlane(aoffs, na);
        const unsigned la = lds0 + (na & 3) * SLOT + W_TILE + 4096;
        dma8k_lin(sb + q * (8 * UNIT), vlin, la);
        if (na < NP1) dma8k_lin(sb + (q ^ 1) * (8 * UNIT), vlin, la + UNIT);
        ++na;
      };
      for (int k = 2; k < NP; ++k) {
        const int lim = k + D + 1 > NP ? NP : k + D + 1;
        if (avail < lim) {
          poll_read();
          if (avail <= k + 1) {
            const u64 t0 = __builtin_amdgcn_s_memrealtime();
            while (true) {
              poll_issue();
              WAIT_V(0);
              poll_read();
              if (avail > k + 1) break;
              if (__builtin_amdgcn_s_memrealtime() - t0 > DEADLINE) { avail = 64; *fl = ~0u; ctl[0] = 1; break; }
              __builtin_amdgcn_s_sleep(1);
            }
          }
        }
        const int top = avail < lim ? avail : lim;
        while (na < top) issue();
        if (avail < lim) poll_issue();
        if (k >= NP1 + 2 && xst < 2) {
          const char* xf = myws + WS_XFLAG + b * 128;
          u64 t1 = 0;
          while (true) {
            if (xst == 1 && __all(fl[64] >= call)) { xst = 2; break; }
            dma4_sys(xf, (lane & 3) * 4, lds0 + LDS_XFL);
            xst = 1;
            if (k < NP - 1) break;
            const u64 now = __builtin_amdgcn_s_memrealtime();
            if (t1 == 0) t1 = now; else __builtin_amdgcn_s_sleep(1);
            if (now - t1 > DEADLINE) { ctl[0] = 1; xst = 2; break; }
            WAIT_V(0);
          }
          if (xst == 2) {
            const char* xs = myws + WS_XCH + (size_t)b * 16384 + 4096;
            dma8k_lin(xs, vlin, lds0 + LDS_X + 4096); dma8k_lin(xs + 8192, vlin, lds0 + LDS_X + 8192 + 4096);
            TS(8);
          }
        }
        const int ahead = na - 2 - k;
        if (k + 3 < NP1) {
          if (ahead >= 2) WAIT_V(32); else if (ahead == 1) WAIT_V(16); else WAIT_V(0);
        } else {
          if (ahead >= 2) WAIT_V(16); else if (ahead == 1) WAIT_V(8); else WAIT_V(0);
        }
        WAIT_L0();
        BAR();
      }
    } else {
      const u64 t_start = __builtin_amdgcn_s_memrealtime();
      const int pslot = b >> 3, pj = pslot >> 2, prb = pslot & 3;
      const unsigned pd = (unsigned)(b + pslot) & 7u;
      if (pd == (unsigned)rank) return;
      const char* ps = p.x + (size_t)(pd * MLOC + prb * 128) * (KLOC * 2) + pj * 128;
      const unsigned stg = lds0 + LDS_X;
      auto stage = [&]() { dma8k_v(ps, voffx, stg + 4096); dma8k_v(ps + 64 * KLOC * 2, voffx, stg + UNIT + 4096); };
      dma4_sys(cnt, 0u, lds0 + LDS_CNT + 256);
      if (pj == 0) stage();
      char* pw = p.ws[pd];
      const u64 t_push = t_start + PUSH0 + pslot * TAU4;
      WAIT_V(0);
      const unsigned call = __builtin_amdgcn_readfirstlane(ctl[128]) + 1;
      char* pdst = pw + (call & 1) * WS_BUF + (size_t)((rank * 8 + prb * 2) * 8 + pj) * UNIT + lane * 16;
      char* pflag = pw + WS_FLAG + (prb * 8 + rank) * 128 + pj * 4;
      bool starving = true;
      if (pj != 0) {
        const int tab = sched_code(lane < NP ? lane : 0, rank, q, rot);
        const int kst = pj <= 2 ? 0 : 2;
        BAR();
        for (int k = 0;; ++k) {
          if (k == kst) stage();
          if (k > kst) {
            const int code = __builtin_amdgcn_readlane(tab, k + 1);
            if (k >= NP1 || (k >= 3 && (code & 7) >= pj)) break;
            bool go = false;
            while (true) {
              starving = k >= 3 && (unsigned)__builtin_amdgcn_readlane(*fl, code) < call;
              if (__builtin_amdgcn_s_memrealtime() + (starving ? 0 : 20) >= t_push) { go = true; break; }
              if (!starving) break;
              __builtin_amdgcn_s_sleep(2);
            }
            if (go) break;
            starving = true;
          }
          BAR();
        }
        WAIT_V(0);
      } else {
        while (__builtin_amdgcn_s_memrealtime() < t_push) __builtin_amdgcn_s_sleep(1);
      }
#pragma unroll
      for (int u = 0; u < 2; ++u) {
        u32x4 pv[8];
#pragma unroll
        for (int i = 0; i < 8; ++i) pv[i] = *(volatile LAS u32x4*)(size_t)(stg + u * UNIT + i * 1024 + lane * 16);
#pragma unroll
        for (int i = 0; i < 8; ++i) st16_sys(pdst + u * 8 * UNIT + i * 1024, pv[i]);
      }
      TS(5);
      if (!starving) BAR();
      WAIT_V(0);
      st4_sys_l0(pflag, call);
      TS(7);
      BAR(); BAR();
    }
  }
}
}

extern "C" void kernel_launch(int rank, int nranks, ncclComm_t comm, hipStream_t stream,
                              void* const* d_in, const int* in_sizes, int n_in,
                              void* d_out, int out_size, const int* in_dtypes, int out_dtype,
                              void* const* d_out_all, void* const* d_ws_all, size_t ws_bytes) {
  if (nranks != NR || ws_bytes < WS_END || (unsigned)rank >= (unsigned)NR) { fprintf(stderr, "bad config\n"); return; }
  static bool init = false;
  if (!init) {
    HIP_CHECK(hipFuncSetAttribute((const void*)a2a_gemm, hipFuncAttributeMaxDynamicSharedMemorySize, LDS_BYTES));
    init = true;
  }
  Params p;
  p.x = (const char*)d_in[0];
  p.w = (const char*)d_in[1];
  p.out = (char*)d_out;
  for (int i = 0; i < NR; ++i) p.ws[i] = (char*)d_ws_all[i];
  p.myws = (char*)d_ws_all[rank];
  p.rank = rank;
  p.pad = 0;
  hipLaunchKernelGGL(a2a_gemm, dim3(256), dim3(448), LDS_BYTES, stream, p);
  HIP_CHECK(hipGetLastError());
}
```

```cpp
#include <hip/hip_runtime.h>
#include <rccl/rccl.h>
#include <cstdio>
#include <type_traits>

#define HIP_CHECK(expr) do { hipError_t _e = (expr); if (_e != hipSuccess) { \
    fprintf(stderr, "HIP error %s at %s:%d\n", hipGetErrorString(_e), __FILE__, __LINE__); \
    return; } } while (0)

typedef unsigned u32x4 __attribute__((ext_vector_type(4)));
typedef float f32x4 __attribute__((ext_vector_type(4)));
typedef __bf16 bf16x4 __attribute__((ext_vector_type(4)));
typedef __bf16 bf16x8 __attribute__((ext_vector_type(8)));
typedef __bf16 bf16x2 __attribute__((ext_vector_type(2)));
typedef unsigned long long u64;
#define LAS __attribute__((address_space(3)))

namespace {
constexpr int NR = 8, MLOC = 512, KLOC = 512, NN = 4096;
constexpr int W_TILE = 128 * 64 * 2;
constexpr int UNIT = 64 * 64 * 2;
constexpr int SLOT = W_TILE + 2 * UNIT;
constexpr int S = 4, D = 3;
#define FSTEPS 8
constexpr int F = FSTEPS;
constexpr int NP1 = (64 - F) / 2;
constexpr int NP = NP1 + F;
static_assert(F % 2 == 0 && F >= 2 && F <= 8, "phase 2 length");
#define XAT 2
constexpr int XB = 6 * 1024;
constexpr int LDS_X = S * SLOT;
constexpr int LDS_CTL = LDS_X + 4 * XB;
constexpr int LDS_CNT = LDS_CTL + 256;
constexpr int LDS_FLG = LDS_CNT + 512;
constexpr int LDS_JNK = LDS_FLG + 256;
constexpr int LDS_BYTES = LDS_JNK + 256;
#define TAU 148
#define PTH 2
#define EARLYN 3
#define KST 2
#define KST2 1
#define KSL 6
#define DA 2
#define VIS 240
#define PEAR 20
#define HOLD 12
#define PUSH0 65
constexpr size_t WS_BUF = (size_t)NR * MLOC * KLOC * 2;
constexpr size_t WS_XCH = 2 * WS_BUF;
constexpr size_t WS_FLAG = WS_XCH + 256 * 4 * XB;
constexpr size_t WS_CNT = WS_FLAG + 256 * 128;
constexpr size_t WS_END = WS_CNT + 256 * 128;
constexpr u64 DEADLINE = 50000000ull;

struct Params {
  const char* x;
  const char* w;
  char* out;
  char* myws;
  int rank;
  int pad;
  char* ws[NR];
};

#define WAIT_V(n) asm volatile("s_waitcnt vmcnt(%0)" ::"n"(n) : "memory")
#define WAIT_L0() __builtin_amdgcn_s_waitcnt(0xc07f)
#define BAR() __builtin_amdgcn_s_barrier()
#define SB() __builtin_amdgcn_sched_barrier(0)

__device__ __forceinline__ void st16_sys(void* p, u32x4 v) {
  asm volatile("global_store_dwordx4 %0, %1, off sc0 sc1\n\ts_nop 1" ::"v"(p), "v"(v) : "memory");
}
__device__ __forceinline__ void st4_sys_l0(void* p, unsigned v) {
  asm volatile("s_mov_b64 exec, 1\n\tglobal_store_dword %0, %1, off sc0 sc1\n\ts_mov_b64 exec, -1" ::"v"(p), "v"(v) : "memory");
}
__device__ __forceinline__ unsigned ld4_sys(const void* p) {
  unsigned v;
  asm volatile("global_load_dword %0, %1, off sc0 sc1\n\ts_waitcnt vmcnt(0)" : "=&v"(v) : "v"(p) : "memory");
  return v;
}
#define GL_(o) "global_load_lds_dwordx4 %0, %1 offset:" #o " sc0 sc1\n\t"
__device__ __forceinline__ void dma8k_lin(const char* sbase, unsigned voff, unsigned ldsaddr) {
  asm volatile("s_mov_b32 m0, %2\n\ts_nop 0\n\t" GL_(-4096) GL_(-3072) GL_(-2048) GL_(-1024) GL_(0) GL_(1024) GL_(2048) GL_(3072)
               ::"v"(voff), "s"(sbase), "s"(ldsaddr) : "memory");
}
#define GV_(i, o) "global_load_lds_dwordx4 %" #i ", %8 offset:" #o "\n\t"
__device__ __forceinline__ void dma8k_v(const char* sbase, const unsigned (&v)[8], unsigned ldsaddr) {
  asm volatile("s_mov_b32 m0, %9\n\ts_nop 0\n\t" GV_(0, -4096) GV_(1, -3072) GV_(2, -2048) GV_(3, -1024) GV_(4, 0) GV_(5, 1024) GV_(6, 2048) GV_(7, 3072)
               ::"v"(v[0]), "v"(v[1]), "v"(v[2]), "v"(v[3]), "v"(v[4]), "v"(v[5]), "v"(v[6]), "v"(v[7]), "s"(sbase), "s"(ldsaddr) : "memory");
}
#define GS_(i, o) "global_store_dwordx4 %8, %" #i ", %9 offset:" #o " sc0 sc1\n\t"
__device__ __forceinline__ void st8k_sys(const char* sbase, unsigned voff, const u32x4* v) {
  asm volatile(GS_(0, -4096) GS_(1, -3072) GS_(2, -2048) GS_(3, -1024) GS_(4, 0) GS_(5, 1024) GS_(6, 2048) GS_(7, 3072) "s_nop 1"
               ::"v"(v[0]), "v"(v[1]), "v"(v[2]), "v"(v[3]), "v"(v[4]), "v"(v[5]), "v"(v[6]), "v"(v[7]), "v"(voff), "s"(sbase) : "memory");
}
#define GR_(i, a, o) "global_load_dwordx4 %" #i ", %" #a ", %16 offset:" #o "\n\t"
__device__ __forceinline__ void ld8k(const char* sbase, const unsigned (&v)[8], u32x4* d) {
  asm volatile(GR_(0, 8, -4096) GR_(1, 9, -3072) GR_(2, 10, -2048) GR_(3, 11, -1024) GR_(4, 12, 0) GR_(5, 13, 1024) GR_(6, 14, 2048) GR_(7, 15, 3072)
               : "=&v"(d[0]), "=&v"(d[1]), "=&v"(d[2]), "=&v"(d[3]), "=&v"(d[4]), "=&v"(d[5]), "=&v"(d[6]), "=&v"(d[7])
               : "v"(v[0]), "v"(v[1]), "v"(v[2]), "v"(v[3]), "v"(v[4]), "v"(v[5]), "v"(v[6]), "v"(v[7]), "s"(sbase) : "memory");
}
#define MFMA(c, x, y) asm volatile("v_mfma_f32_16x16x32_bf16 %0, %1, %2, %0" : "+v"(c) : "v"(x), "v"(y))
__device__ __forceinline__ void dma4_sys(const char* sbase, unsigned voff, unsigned ldsaddr) {
  asm volatile("s_mov_b32 m0, %2\n\ts_nop 0\n\tglobal_load_lds_dword %0, %1 sc0 sc1" ::"v"(voff), "s"(sbase), "s"(ldsaddr) : "memory");
}
__device__ __forceinline__ void dma4(const char* sbase, unsigned voff, unsigned ldsaddr) {
  asm volatile("s_mov_b32 m0, %2\n\ts_nop 0\n\tglobal_load_lds_dword %0, %1" ::"v"(voff), "s"(sbase), "s"(ldsaddr) : "memory");
}
__device__ __forceinline__ int first_clear(bool c) {
  const u64 m = ~__ballot(c);
  return __builtin_amdgcn_readfirstlane(m ? __builtin_ctzll(m) : 64);
}
__device__ __forceinline__ unsigned pk(float a, float b) { return __builtin_bit_cast(unsigned, (bf16x2){(__bf16)a, (__bf16)b}); }

__device__ __forceinline__ int sched_code(int e, int rank, int q, int rot, int phi) {
  if (e < 4) return rank * 8 + 2 * ((e + rot) & 3) + q;
  int u;
  if (e < NP1) {
    const int u0 = 2 * (e - 4) + q;
    const int r = u0 / 7;
    const int first = 7 * r + ((7 * r + q) & 1);
    int last = 7 * r + 6; if (last > 55 - F) last = 55 - F;
    const int c = (last - first) / 2 + 1;
    u = first + 2 * (((u0 - first) / 2 + rot) % c);
  } else {
    u = 56 - F + ((e - NP1 + rot * 2 + q) % F);
  }
  const int r = u / 7, i = u - 7 * r;
  return ((rank + 1 + i) & 7) * 8 + r;
}

#define TC(i) do {} while (0)
#define TS(i) do {} while (0)

struct Fr { bf16x8 a[2][4]; bf16x4 b[2][4][2]; };

__global__ __launch_bounds__(448, 2) void a2a_gemm(Params p) {
  extern __shared__ __attribute__((aligned(1024))) char smem[];
#define STAMP(x) do {} while (0)
  const unsigned lds0 = (unsigned)(size_t)(LAS char*)smem;
  const int b = blockIdx.x;
  const int wid = __builtin_amdgcn_readfirstlane(threadIdx.x >> 6);
  const int lane = threadIdx.x & 63;
  const int xcd = b & 7, idx = b >> 3;
  const int q = xcd >> 2, bn = (xcd & 3) * 8 + (idx & 7), bm2 = idx >> 3;
  asm volatile("" : "+s"(p.x), "+s"(p.w), "+s"(p.out), "+s"(p.myws), "+s"(p.rank), "+s"(p.ws[0]), "+s"(p.ws[1]), "+s"(p.ws[2]), "+s"(p.ws[3]),
               "+s"(p.ws[4]), "+s"(p.ws[5]), "+s"(p.ws[6]), "+s"(p.ws[7]));
  const int rank = p.rank;
  const int rot = bm2;
  char* myws = p.myws;
  const unsigned pd = (unsigned)(b + idx) & 7u;
  const bool early = idx < EARLYN && pd != (unsigned)rank;
  volatile LAS unsigned* ctl = (volatile LAS unsigned*)(size_t)(lds0 + LDS_CTL);

  if (wid < 4) {
    if (wid == 0) TS(0);
    const int wm = wid >> 1, wn = wid & 1;
    const int l15 = lane & 15, g = lane >> 4, qq = l15 >> 2, pp = lane & 3;
    const unsigned aoff = W_TILE + (wm * 32 + l15) * 128 + ((g ^ (l15 >> 1)) << 4);
    const unsigned boff = (8 * g + qq) * 256 + (((wn * 8 + (pp >> 1)) ^ (qq << 2) ^ ((2 * g) & 3)) << 4) + 8 * (pp & 1);
    f32x4 acc[4][4];
#pragma unroll
    for (int m = 0; m < 4; ++m)
#pragma unroll
      for (int n = 0; n < 4; ++n) acc[m][n] = (f32x4){0.f, 0.f, 0.f, 0.f};
    unsigned ra[2][2], rb[2][8];
#pragma unroll
    for (int hh = 0; hh < 2; ++hh) {
#pragma unroll
      for (int k2 = 0; k2 < 2; ++k2) { ra[hh][k2] = lds0 + hh * 65536 + (aoff ^ (k2 << 6)); asm volatile("" : "+v"(ra[hh][k2])); }
#pragma unroll
      for (int u = 0; u < 8; ++u) { rb[hh][u] = lds0 + hh * 65536 + (boff ^ (u << 4)); asm volatile("" : "+v"(rb[hh][u])); }
    }
    auto load = [&](Fr& f, auto slc, auto mc) {
      constexpr int so = decltype(slc)::value * SLOT, hh = so / 65536, o = so % 65536, MT = decltype(mc)::value;
#pragma unroll
      for (int k2 = 0; k2 < 2; ++k2)
#pragma unroll
        for (int n = 0; n < 4; ++n) {
#pragma unroll
          for (int t = 0; t < 2; ++t)
            f.b[k2][n][t] = __builtin_amdgcn_ds_read_tr16_b64_v4bf16((LAS bf16x4*)(size_t)(rb[hh][n * 2 + t] + o + t * 1024 + k2 * 8192));
          if (n < MT) f.a[k2][n] = *(const LAS bf16x8*)(size_t)(ra[hh][k2] + o + (n >> 1) * UNIT + (n & 1) * 2048);
        }
    };
    auto period = [&](Fr& cur, Fr& nxt, auto slc, auto mcur, auto mnxt) {
      constexpr int so = decltype(slc)::value * SLOT, hh = so / 65536, o = so % 65536;
      constexpr int MC = decltype(mcur)::value, MN = decltype(mnxt)::value;
      constexpr int NM = MC * 8, NRD = MN ? 16 + MN * 2 : 0, PER = MC == 4 ? 1 : 2;
      SB();
#pragma unroll
      for (int i = 0; i < NM; ++i) {
        const int n = i / (MC * 2), m = (i >> 1) % MC, k2 = i & 1;
        bf16x8 wv = __builtin_shufflevector(cur.b[k2][n][0], cur.b[k2][n][1], 0, 1, 2, 3, 4, 5, 6, 7);
        MFMA(acc[m][n], wv, cur.a[k2][m]);
#pragma unroll
        for (int rr = i * PER; rr < i * PER + PER; ++rr) {
          if (rr < NRD) {
            if (rr >= 4 && rr < 4 + MN * 2) {
              const int mm = (rr - 4) >> 1, kk = rr & 1;
              nxt.a[kk][mm] = *(const LAS bf16x8*)(size_t)(ra[hh][kk] + o + (mm >> 1) * UNIT + (mm & 1) * 2048);
            } else {
              const int x = rr < 4 ? rr : rr - MN * 2, nn = x >> 2, kk = (x >> 1) & 1, t = x & 1;
              nxt.b[kk][nn][t] = __builtin_amdgcn_ds_read_tr16_b64_v4bf16((LAS bf16x4*)(size_t)(rb[hh][nn * 2 + t] + o + t * 1024 + kk * 8192));
            }
          }
        }
        SB();
      }
    };
    using I0 = std::integral_constant<int, 0>; using I1 = std::integral_constant<int, 1>;
    using I2 = std::integral_constant<int, 2>; using I3 = std::integral_constant<int, 3>; using I4 = std::integral_constant<int, 4>;
#define ENDP() do { WAIT_L0(); BAR(); } while (0)
    Fr f0, f1;
    if (early) BAR();
    BAR();
    if (wid == 0) { TS(3); TC(15); TS(15); }
    load(f0, I0{}, I4{});
    const unsigned call = __builtin_amdgcn_readfirstlane(ctl[64]) + 1;
    ENDP();
    constexpr int NPL = (NP1 - 1) / 4 * 4;
    for (int pr = 0; pr < NPL; pr += 4) {
      period(f0, f1, I1{}, I4{}, I4{}); ENDP();
      period(f1, f0, I2{}, I4{}, I4{}); ENDP();
      period(f0, f1, I3{}, I4{}, I4{}); ENDP();
      period(f1, f0, I0{}, I4{}, I4{}); ENDP();
    }
    const unsigned tagh = call << 16;
    const char* xsrc = myws + WS_XCH + (size_t)(b * 4 + wid) * XB + 4096;
    const unsigned xl = lds0 + LDS_X + wid * XB;
    auto xfetch = [&]() {
      asm volatile("s_mov_b32 m0, %2\n\ts_nop 0\n\t" GL_(-4096) GL_(-3072) GL_(-2048) GL_(-1024) GL_(0) GL_(1024) ::"v"(lane * 16), "s"(xsrc), "s"(xl + 4096) : "memory");
    };
    auto tail = [&](auto self, auto pc) {
      constexpr int P = decltype(pc)::value;
      if constexpr (P == NP1) {
        asm volatile("s_nop 7" : "+v"(acc[3][3]), "+v"(acc[2][3]));
        if (wid == 0) { TS(6); TC(22); TS(22); }
        char* xd = myws + WS_XCH + (size_t)((b ^ 4) * 4 + wid) * XB + lane * 16;
        auto val = [&](int i) { return i < 32 ? acc[2 + (i >> 4)][(i >> 2) & 3][i & 3] : 0.f; };
#pragma unroll
        for (int i = 0; i < 6; ++i) {
          const unsigned t = pk(val(6 * i + 2), val(6 * i + 5));
          st16_sys(xd + i * 1024, (u32x4){pk(val(6 * i), val(6 * i + 1)), (t & 0xffffu) | tagh, pk(val(6 * i + 3), val(6 * i + 4)), (t >> 16) | tagh});
        }
      }
      if constexpr (P == NP - XAT) xfetch();
      using SLc = std::integral_constant<int, (P + 1) & 3>;
      using MCc = std::integral_constant<int, (P < NP1) ? 4 : 2>;
      using MNc = std::integral_constant<int, (P + 1 < NP1) ? 4 : (P + 1 < NP) ? 2 : 0>;
      if constexpr (P & 1) period(f1, f0, SLc{}, MCc{}, MNc{}); else period(f0, f1, SLc{}, MCc{}, MNc{});
      if constexpr (P + 2 < NP) ENDP();
      if constexpr (P + 1 < NP) self(self, std::integral_constant<int, P + 1>{});
    };
    tail(tail, std::integral_constant<int, NPL>{});
    bool fail = __builtin_amdgcn_readfirstlane(ctl[0]) != 0;
    u32x4 xp[6];
    {
      u64 t0 = 0;
      while (true) {
        WAIT_V(0);
        unsigned bad = 0;
#pragma unroll
        for (int i = 0; i < 6; ++i) { xp[i] = *(volatile LAS u32x4*)(size_t)(xl + i * 1024 + lane * 16); bad |= (xp[i][1] ^ tagh) | (xp[i][3] ^ tagh); }
        if (__all(bad < 0x10000u) || fail) break;
        const u64 now = __builtin_amdgcn_s_memrealtime();
        if (t0 == 0) t0 = now;
        if (now - t0 > DEADLINE) fail = true;
        xfetch();
      }
    }
    asm volatile("s_nop 7\n\ts_nop 1" : "+v"(acc[1][3]), "+v"(acc[0][3]), "+v"(acc[1][2]));
    if (wid == 0) { TS(1); TC(23); TS(23); }
    char* obase = p.out + ((size_t)((bm2 * 2 + q) * 64 + wm * 32 + l15) * NN + bn * 128 + wn * 64) * 2 + ((g & 1) * 16 + (g >> 1) * 8) * 2;
#pragma unroll
    for (int m = 0; m < 2; ++m)
#pragma unroll
      for (int n = 0; n < 4; n += 2) {
        const f32x4 c0 = acc[m][n], c1 = acc[m][n + 1];
        auto xv = [&](int i) {
          const int r6 = i % 6, w = r6 < 2 ? 0 : r6 == 2 ? 1 : r6 < 5 ? 2 : 3;
          const unsigned v = xp[i / 6][w];
          return __uint_as_float(r6 == 1 || r6 == 4 ? v & 0xffff0000u : v << 16);
        };
        const int i0 = m * 16 + n * 4;
        const unsigned a0 = pk(c0[0] + xv(i0), c0[1] + xv(i0 + 1)), a1 = pk(c0[2] + xv(i0 + 2), c0[3] + xv(i0 + 3));
        const unsigned b0 = pk(c1[0] + xv(i0 + 4), c1[1] + xv(i0 + 5)), b1 = pk(c1[2] + xv(i0 + 6), c1[3] + xv(i0 + 7));
        auto r0 = __builtin_amdgcn_permlane16_swap(a0, b0, false, false);
        auto r1 = __builtin_amdgcn_permlane16_swap(a1, b1, false, false);
        u32x4 v = {r0[0], r1[0], r0[1], r1[1]};
        if (fail) v = (u32x4){0x7fc07fc0u, 0x7fc07fc0u, 0x7fc07fc0u, 0x7fc07fc0u};
        st16_sys(obase + (size_t)m * 16 * NN * 2 + n * 32, v);
      }
  } else if (wid == 4) {
    const int r = lane >> 4, c = lane & 15;
    unsigned voff[8];
#pragma unroll
    for (int u = 0; u < 8; ++u) voff[u] = (4 * u + r) * (NN * 2) + ((c ^ (r << 2) ^ (u & 3)) << 4) + 4096 - u * 1024;
    const char* base = p.w + bn * 256;
    auto issue = [&](const char* sb, unsigned la) { dma8k_v(sb, voff, la); dma8k_v(sb + 32 * NN * 2, voff, la + 8192); };
    auto src = [&](int code) { return base + (size_t)code * (64 * NN * 2); };
    if (early) BAR(); else __builtin_amdgcn_s_sleep(HOLD);
    issue(src(rank * 8 + 2 * (rot & 3) + q), lds0 + 4096);
    issue(src(rank * 8 + 2 * ((1 + rot) & 3) + q), lds0 + 4096 + SLOT);
    const int tab = sched_code(lane < NP ? lane : 0, rank, q, rot, bn & 3);
    WAIT_V(16);
    BAR();
    issue(src(rank * 8 + 2 * ((2 + rot) & 3) + q), lds0 + 4096 + 2 * SLOT);
    WAIT_V(16);
    BAR();
    issue(src(rank * 8 + 2 * ((3 + rot) & 3) + q), lds0 + 4096 + 3 * SLOT);
    const char* sb = src(__builtin_amdgcn_readlane(tab, 4));
    unsigned la = lds0 + 4096;
    for (int k = 1; k < NP - D; ++k) {
      dma8k_v(sb, voff, la);
      const char* sb2 = sb + 32 * NN * 2;
      unsigned la2 = la + 8192;
      sb = src(__builtin_amdgcn_readlane(tab, k + D + 1));
      la = lds0 + 4096 + ((k + D + 1) & 3) * SLOT;
      asm volatile("" : "+s"(sb), "+s"(la), "+s"(sb2), "+s"(la2));
      dma8k_v(sb2, voff, la2);
      WAIT_V(32);
      BAR();
    }
    WAIT_V(16); BAR();
    WAIT_V(0); BAR();
  } else {
    const int r = lane >> 3, c = lane & 7;
    unsigned voffx[8];
#pragma unroll
    for (int u = 0; u < 8; ++u) voffx[u] = (8 * u + r) * (KLOC * 2) + ((c ^ (((u & 1) << 2) | (r >> 1))) << 4) + 4096 - u * 1024;
    char* cnt = myws + WS_CNT + b * 128;
    volatile LAS unsigned* fl = (volatile LAS unsigned*)(size_t)(lds0 + LDS_FLG + lane * 4);
    if (wid == 5) {
      const u64 t_start = __builtin_amdgcn_s_memrealtime();
      const char* xb = p.x + (size_t)(rank * MLOC + bm2 * 128) * (KLOC * 2);
      auto issue_local = [&](int e) {
        const char* sb = xb + (2 * ((e + rot) & 3) + q) * 128;
        const unsigned la = lds0 + e * SLOT + W_TILE + 4096;
        dma8k_v(sb + q * (64 * KLOC * 2), voffx, la);
        dma8k_v(sb + (q ^ 1) * (64 * KLOC * 2), voffx, la + UNIT);
      };
      fl[0] = 0; ctl[0] = 0;
      if (early) BAR(); else __builtin_amdgcn_s_sleep(HOLD);
      dma4_sys(cnt, 0u, lds0 + LDS_CNT);
      issue_local(0); issue_local(1);
      const int tab = sched_code(lane < NP ? lane : 0, rank, q, rot, bn & 3);
      WAIT_V(16);
      const unsigned call = __builtin_amdgcn_readfirstlane(ctl[64]) + 1;
      const char* db = myws + (call & 1) * WS_BUF + (size_t)(bm2 * 2) * 8 * UNIT + 4096;
      const char* fp = myws + WS_FLAG + bm2 * 8192;
      const unsigned vlin = lane * 16;
      const bool nocheck = lane < 4 || lane >= NP;
      const unsigned fvo = tab * 128;
      const unsigned texp = PUSH0 + VIS + (((tab & 7) * 4 + ((((((tab >> 3) - rank - 1) & 7) >> 1) - bm2) & 3)) * TAU >> 2);
      const unsigned aoffs = (tab >> 3) * (MLOC * KLOC * 2) + (tab & 7) * UNIT;
      int avail = 4;
      int since = 0;
      const unsigned fthr = nocheck ? 0u : call;
      const unsigned tlow = (unsigned)t_start;
      auto poll_fire = [&](int upto, unsigned now) {
        const u64 m = __ballot((lane >= avail) & (lane < NP) & ((lane <= upto) | (texp <= now - tlow)));
        asm volatile("s_mov_b32 m0, %2\n\ts_mov_b64 exec, %3\n\tglobal_load_lds_dword %0, %1 sc0 sc1\n\ts_mov_b64 exec, -1" ::"v"(fvo), "s"(fp), "s"(lds0 + LDS_FLG), "s"(m) : "memory");
        ++since;
      };
      auto poll_issue = [&](int upto) { poll_fire(upto, (unsigned)__builtin_amdgcn_s_memrealtime()); };
      auto poll_read = [&]() { avail = first_clear(*fl >= fthr); };
      WAIT_L0();
      BAR();
      issue_local(2);
      WAIT_V(16);
      BAR();
      issue_local(3);
      WAIT_V(16);
      BAR();
      st4_sys_l0(cnt, call);
      ++since;
      int na = 4;
      auto issue = [&]() {
        const char* sb = db + (unsigned)__builtin_amdgcn_readlane(aoffs, na);
        const unsigned la = lds0 + (na & 3) * SLOT + W_TILE + 4096;
        dma8k_lin(sb + q * (8 * UNIT), vlin, la);
        if (na < NP1) dma8k_lin(sb + (q ^ 1) * (8 * UNIT), vlin, la + UNIT);
        ++na; since = 0;
      };
      auto gate = [&](int k) {
        if (avail < k + DA + 1 + PTH && avail < NP) {
          poll_read();
          if (avail <= k + 1) {
            const u64 t0 = __builtin_amdgcn_s_memrealtime();
            while (true) {
              poll_issue(k + 2);
              WAIT_V(0);
              poll_read();
              if (avail > k + 1) break;
              if (__builtin_amdgcn_s_memrealtime() - t0 > DEADLINE) { avail = 64; *fl = ~0u; ctl[0] = 1; WAIT_L0(); break; }
              __builtin_amdgcn_s_sleep(1);
            }
          }
          if (avail < k + DA + 1 + PTH && avail < NP) poll_issue(avail);
        }
      };
      int kpoll = 0;
      const char* db0 = db + q * (8 * UNIT);
      const char* db1 = db + (q ^ 1) * (8 * UNIT);
      for (int k = 2; k < NP1 - DA; ++k) {
        if (na == k + DA && avail > na) {
          const unsigned o = __builtin_amdgcn_readlane(aoffs, na);
          const unsigned la = lds0 + (na & 3) * SLOT + W_TILE + 4096;
          const char* s0 = db0 + o; const char* s1 = db1 + o;
          unsigned la1 = la + UNIT;
          ++na;
          asm volatile("" : "+s"(s1), "+s"(la1), "+s"(na));
          if (k < kpoll) {
            dma8k_lin(s0, vlin, la);
            SB();
            const unsigned fv = *fl;
            SB();
            dma8k_lin(s1, vlin, la1);
            WAIT_V(16 * (DA - 1));
            BAR();
            avail = first_clear(fv >= fthr);
          } else if (avail < k + DA + 1 + PTH) {
            const unsigned now = (unsigned)__builtin_amdgcn_s_memrealtime();
            SB();
            dma8k_lin(s0, vlin, la);
            SB();
            poll_fire(avail, now); kpoll = k + 2;
            SB();
            dma8k_lin(s1, vlin, la1);
            WAIT_V(16 * (DA - 1) + 1);
            BAR();
          } else {
            dma8k_lin(s0, vlin, la);
            dma8k_lin(s1, vlin, la1);
            WAIT_V(16 * (DA - 1));
            BAR();
          }
          since = 0;
          continue;
        } else {
          gate(k);
          kpoll = k + 2;
          const int lim = k + DA + 1;
          const int top = avail < lim ? avail : lim;
          while (na < top) issue();
          const int ahead = na - 2 - k;
          if (ahead >= 2) WAIT_V(32); else if (ahead == 1) WAIT_V(16);
          else if (since >= 3) WAIT_V(3); else if (since == 2) WAIT_V(2); else if (since == 1) WAIT_V(1); else WAIT_V(0);
        }
        BAR();
      }
      for (int k = NP1 - DA; k < NP - 1; ++k) {
        const int lim = k + D + 1 > NP ? NP : k + D + 1;
        gate(k);
        const int top = avail < lim ? avail : lim;
        while (na < top) issue();
        const int ahead = na - 2 - k;
        if (ahead >= 2) WAIT_V(16); else if (ahead == 1) WAIT_V(8); else WAIT_V(0);
        BAR();
      }
    } else {
      const u64 t_start = __builtin_amdgcn_s_memrealtime();
      const int pj = idx >> 2;
      const int prb = ((((rank - (int)pd - 1) & 7) >> 1) - idx) & 3;
      if (pd == (unsigned)rank) return;
      const char* ps = p.x + (size_t)(pd * MLOC + prb * 128) * (KLOC * 2) + pj * 128;
      const unsigned stg = lds0 + LDS_X;
      auto stage = [&]() { dma8k_v(ps, voffx, stg + 4096); dma8k_v(ps + 64 * KLOC * 2, voffx, stg + UNIT + 4096); };
      u32x4 pv[16];
      unsigned call;
      if (pj <= 1) {
        unsigned cv;
        STAMP(st1);
        ld8k(ps, voffx, pv);
        asm volatile("global_load_dword %0, %1, %2 sc0 sc1" : "=v"(cv) : "v"(0u), "s"(cnt) : "memory");
        ld8k(ps + 64 * KLOC * 2, voffx, pv + 8);
        STAMP(st2);
        asm volatile("s_waitcnt vmcnt(0)" : "+v"(cv), "+v"(pv[0]), "+v"(pv[1]), "+v"(pv[2]), "+v"(pv[3]), "+v"(pv[4]), "+v"(pv[5]), "+v"(pv[6]), "+v"(pv[7]),
                     "+v"(pv[8]), "+v"(pv[9]), "+v"(pv[10]), "+v"(pv[11]), "+v"(pv[12]), "+v"(pv[13]), "+v"(pv[14]), "+v"(pv[15]) :: "memory");
        STAMP(st3);
        call = __builtin_amdgcn_readfirstlane(cv) + 1;
      } else {
        BAR();
        call = __builtin_amdgcn_readfirstlane(ctl[64]) + 1;
      }
      char* pw = p.ws[0];
#pragma unroll
      for (int i = 1; i < NR; ++i) pw = pd == (unsigned)i ? p.ws[i] : pw;
      const u64 t_push = t_start + PUSH0 + ((idx * TAU) >> 2);
      const char* pd0 = pw + (call & 1) * WS_BUF + (size_t)((rank * 8 + prb * 2) * 8 + pj) * UNIT + 4096;
      char* pflag = pw + WS_FLAG + ((prb * 8 + rank) * 8 + pj) * 128;
      bool starving = true;
      if (pj != 0) {
        const int tab = sched_code(lane < NP ? lane : 0, rank, q, rot, bn & 3);
        if (pj == 1) BAR();
        int k = 0;
        auto run_until = [&](u64 t) {
          for (;; ++k) {
            starving = true;
            const int code = __builtin_amdgcn_readlane(tab, k + 1);
            if (k >= NP1 - 2 || (k >= 3 && (code & 7) >= pj)) break;
            bool go = false;
            while (true) {
              starving = k >= 3 && (unsigned)__builtin_amdgcn_readlane(*fl, k + 1) < call;
              if (__builtin_amdgcn_s_memrealtime() + (starving ? 0 : PEAR) >= t) { go = true; break; }
              if (!starving) break;
              __builtin_amdgcn_s_sleep(2);
            }
            if (go) break;
            BAR();
          }
        };
        if (pj >= 2) {
          const int kst = pj == 2 ? KST2 : KST;
          for (; k < kst; ++k) BAR();
          __builtin_amdgcn_s_sleep(KSL);
          stage();
          if (k < 3) { BAR(); ++k; }
          WAIT_V(0);
#pragma unroll
          for (int i = 0; i < 16; ++i) pv[i] = *(volatile LAS u32x4*)(size_t)(stg + i * 1024 + lane * 16);
        }
        run_until(t_push);
      } else {
        while (__builtin_amdgcn_s_memrealtime() < t_push) __builtin_amdgcn_s_sleep(1);
      }
      STAMP(st4);
      st8k_sys(pd0, lane * 16, pv);
      st8k_sys(pd0 + 8 * UNIT, lane * 16, pv + 8);
      TS(5);
      if (!starving || early) BAR();
      WAIT_V(0);
      asm volatile("s_mov_b64 exec, 0xffffffff\n\tglobal_store_dword %0, %1, off sc0 sc1\n\ts_mov_b64 exec, -1" ::"v"(pflag + lane * 4), "v"(call) : "memory");
      TS(7);
      BAR(); BAR();
    }
  }
}
}

extern "C" void kernel_launch(int rank, int nranks, ncclComm_t comm, hipStream_t stream,
                              void* const* d_in, const int* in_sizes, int n_in,
                              void* d_out, int out_size, const int* in_dtypes, int out_dtype,
                              void* const* d_out_all, void* const* d_ws_all, size_t ws_bytes) {
  if (nranks != NR || ws_bytes < WS_END || (unsigned)rank >= (unsigned)NR) { fprintf(stderr, "bad config\n"); return; }
  static bool init = false;
  if (!init) {
    HIP_CHECK(hipFuncSetAttribute((const void*)a2a_gemm, hipFuncAttributeMaxDynamicSharedMemorySize, LDS_BYTES));
    init = true;
  }
  Params p;
  p.x = (const char*)d_in[0];
  p.w = (const char*)d_in[1];
  p.out = (char*)d_out;
  for (int i = 0; i < NR; ++i) p.ws[i] = (char*)d_ws_all[i];
  p.myws = (char*)d_ws_all[rank];
  p.rank = rank;
  p.pad = 0;
  hipLaunchKernelGGL(a2a_gemm, dim3(256), dim3(448), LDS_BYTES, stream, p);
  HIP_CHECK(hipGetLastError());
}
```

```cpp
#include <hip/hip_runtime.h>
#include <rccl/rccl.h>
#include <cstdio>
#include <type_traits>

#define HIP_CHECK(expr) do { hipError_t _e = (expr); if (_e != hipSuccess) { \
    fprintf(stderr, "HIP error %s at %s:%d\n", hipGetErrorString(_e), __FILE__, __LINE__); \
    return; } } while (0)

typedef unsigned u32x4 __attribute__((ext_vector_type(4)));
typedef float f32x4 __attribute__((ext_vector_type(4)));
typedef __bf16 bf16x4 __attribute__((ext_vector_type(4)));
typedef __bf16 bf16x8 __attribute__((ext_vector_type(8)));
typedef __bf16 bf16x2 __attribute__((ext_vector_type(2)));
typedef unsigned long long u64;
#define LAS __attribute__((address_space(3)))

namespace {
constexpr int NR = 8, MLOC = 512, KLOC = 512, NN = 4096;
constexpr int W_TILE = 128 * 64 * 2;
constexpr int UNIT = 64 * 64 * 2;
constexpr int SLOT = W_TILE + 2 * UNIT;
constexpr int S = 4, D = 3;
#define FSTEPS 8
constexpr int F = FSTEPS;
constexpr int NP1 = (64 - F) / 2;
constexpr int NP = NP1 + F;
static_assert(F % 2 == 0 && F >= 2 && F <= 8, "phase 2 length");
#define XAT 2
constexpr int XB = 6 * 1024;
constexpr int LDS_X = S * SLOT;
constexpr int LDS_CTL = LDS_X + 4 * XB;
constexpr int LDS_CNT = LDS_CTL + 256;
constexpr int LDS_FLG = LDS_CNT + 512;
constexpr int LDS_JNK = LDS_FLG + 256;
constexpr int LDS_BYTES = LDS_JNK + 256;
#define TAU 148
#define PTH 3
#define EARLYN 3
#define KST 0
#define PJL 9
#define PJM 6
#define KSL 0
#define RUP 1
#define FP0 5
#define DA 2
#define VIS 200
#define TAU1 132
#define NS1 8
#define PE1 50
__device__ __forceinline__ int slot_time(int s) { return s < NS1 ? (s * TAU1 >> 2) : ((NS1 * TAU1 + (s - NS1) * TAU) >> 2); }
#define PEAR 20
#define HOLD 12
#define PUSH0 65
constexpr size_t WS_BUF = (size_t)NR * MLOC * KLOC * 2;
constexpr size_t WS_XCH = 2 * WS_BUF;
constexpr size_t WS_FLAG = WS_XCH + 256 * 4 * XB;
constexpr size_t WS_CNT = WS_FLAG + 256 * 128;
constexpr size_t WS_END = WS_CNT + 256 * 128;
constexpr u64 DEADLINE = 50000000ull;

struct Params {
  const char* x;
  const char* w;
  char* out;
  char* myws;
  int rank;
  int pad;
  char* ws[NR];
};

#define WAIT_V(n) asm volatile("s_waitcnt vmcnt(%0)" ::"n"(n) : "memory")
#define WAIT_L0() __builtin_amdgcn_s_waitcnt(0xc07f)
#define BAR() __builtin_amdgcn_s_barrier()
#define SB() __builtin_amdgcn_sched_barrier(0)

__device__ __forceinline__ void st16_sys(void* p, u32x4 v) {
  asm volatile("global_store_dwordx4 %0, %1, off sc0 sc1\n\ts_nop 1" ::"v"(p), "v"(v) : "memory");
}
__device__ __forceinline__ void st4_sys_l0(void* p, unsigned v) {
  asm volatile("s_mov_b64 exec, 1\n\tglobal_store_dword %0, %1, off sc0 sc1\n\ts_mov_b64 exec, -1" ::"v"(p), "v"(v) : "memory");
}
__device__ __forceinline__ unsigned ld4_sys(const void* p) {
  unsigned v;
  asm volatile("global_load_dword %0, %1, off sc0 sc1\n\ts_waitcnt vmcnt(0)" : "=&v"(v) : "v"(p) : "memory");
  return v;
}
#define APOL " sc0 sc1"
#define GL_(o) "global_load_lds_dwordx4 %0, %1 offset:" #o APOL "\n\t"
__device__ __forceinline__ void dma8k_lin(const char* sbase, unsigned voff, unsigned ldsaddr) {
  asm volatile("s_mov_b32 m0, %2\n\ts_nop 0\n\t" GL_(-4096) GL_(-3072) GL_(-2048) GL_(-1024) GL_(0) GL_(1024) GL_(2048) GL_(3072)
               ::"v"(voff), "s"(sbase), "s"(ldsaddr) : "memory");
}
#define WPOL ""
#define GV_(i, o) "global_load_lds_dwordx4 %" #i ", %8 offset:" #o WPOL "\n\t"
__device__ __forceinline__ void dma8k_v(const char* sbase, const unsigned (&v)[8], unsigned ldsaddr) {
  asm volatile("s_mov_b32 m0, %9\n\ts_nop 0\n\t" GV_(0, -4096) GV_(1, -3072) GV_(2, -2048) GV_(3, -1024) GV_(4, 0) GV_(5, 1024) GV_(6, 2048) GV_(7, 3072)
               ::"v"(v[0]), "v"(v[1]), "v"(v[2]), "v"(v[3]), "v"(v[4]), "v"(v[5]), "v"(v[6]), "v"(v[7]), "s"(sbase), "s"(ldsaddr) : "memory");
}
#define GS_(i, o) "global_store_dwordx4 %8, %" #i ", %9 offset:" #o " sc0 sc1\n\t"
__device__ __forceinline__ void st8k_sys(const char* sbase, unsigned voff, const u32x4* v) {
  asm volatile(GS_(0, -4096) GS_(1, -3072) GS_(2, -2048) GS_(3, -1024) GS_(4, 0) GS_(5, 1024) GS_(6, 2048) GS_(7, 3072) "s_nop 1"
               ::"v"(v[0]), "v"(v[1]), "v"(v[2]), "v"(v[3]), "v"(v[4]), "v"(v[5]), "v"(v[6]), "v"(v[7]), "v"(voff), "s"(sbase) : "memory");
}
#define GR_(i, a, o) "global_load_dwordx4 %" #i ", %" #a ", %16 offset:" #o "\n\t"
__device__ __forceinline__ void ld8k(const char* sbase, const unsigned (&v)[8], u32x4* d) {
  asm volatile(GR_(0, 8, -4096) GR_(1, 9, -3072) GR_(2, 10, -2048) GR_(3, 11, -1024) GR_(4, 12, 0) GR_(5, 13, 1024) GR_(6, 14, 2048) GR_(7, 15, 3072)
               : "=&v"(d[0]), "=&v"(d[1]), "=&v"(d[2]), "=&v"(d[3]), "=&v"(d[4]), "=&v"(d[5]), "=&v"(d[6]), "=&v"(d[7])
               : "v"(v[0]), "v"(v[1]), "v"(v[2]), "v"(v[3]), "v"(v[4]), "v"(v[5]), "v"(v[6]), "v"(v[7]), "s"(sbase) : "memory");
}
#define MFMA(c, x, y) asm volatile("v_mfma_f32_16x16x32_bf16 %0, %1, %2, %0" : "+v"(c) : "v"(x), "v"(y))
__device__ __forceinline__ void dma4_sys(const char* sbase, unsigned voff, unsigned ldsaddr) {
  asm volatile("s_mov_b32 m0, %2\n\ts_nop 0\n\tglobal_load_lds_dword %0, %1 sc0 sc1" ::"v"(voff), "s"(sbase), "s"(ldsaddr) : "memory");
}
__device__ __forceinline__ void dma4(const char* sbase, unsigned voff, unsigned ldsaddr) {
  asm volatile("s_mov_b32 m0, %2\n\ts_nop 0\n\tglobal_load_lds_dword %0, %1" ::"v"(voff), "s"(sbase), "s"(ldsaddr) : "memory");
}
__device__ __forceinline__ int first_clear(bool c) {
  const u64 m = ~__ballot(c);
  return __builtin_amdgcn_readfirstlane(m ? __builtin_ctzll(m) : 64);
}
__device__ __forceinline__ unsigned pk(float a, float b) { return __builtin_bit_cast(unsigned, (bf16x2){(__bf16)a, (__bf16)b}); }

__device__ __forceinline__ int sched_code(int e, int rank, int q, int rot, int phi) {
  if (e < 4) return rank * 8 + 2 * ((e + rot) & 3) + q;
  int u;
  if (e < NP1) {
    const int u0 = 2 * (e - 4) + q;
    const int r = u0 / 7;
    const int first = 7 * r + ((7 * r + q) & 1);
    int last = 7 * r + 6; if (last > 55 - F) last = 55 - F;
    const int c = (last - first) / 2 + 1;
    u = first + 2 * (((u0 - first) / 2 + rot) % c);
  } else {
    u = 56 - F + ((e - NP1 + rot * 2 + q) % F);
  }
  const int r = u / 7, i = u - 7 * r;
  return ((rank + 1 + i) & 7) * 8 + r;
}

#define TC(i) do {} while (0)
#define TS(i) do {} while (0)

struct Fr { bf16x8 a[2][4]; bf16x4 b[2][4][2]; };

__global__ __launch_bounds__(448, 2) void a2a_gemm(Params p) {
  extern __shared__ __attribute__((aligned(1024))) char smem[];
#define STAMP(x) do {} while (0)
  const unsigned lds0 = (unsigned)(size_t)(LAS char*)smem;
  const int b = blockIdx.x;
  const int wid = __builtin_amdgcn_readfirstlane(threadIdx.x >> 6);
  const int lane = threadIdx.x & 63;
  const int xcd = b & 7, idx = b >> 3;
  const int q = xcd >> 2, bn = (xcd & 3) * 8 + (idx & 7), bm2 = idx >> 3;
  asm volatile("" : "+s"(p.x), "+s"(p.w), "+s"(p.out), "+s"(p.myws), "+s"(p.rank), "+s"(p.ws[0]), "+s"(p.ws[1]), "+s"(p.ws[2]), "+s"(p.ws[3]),
               "+s"(p.ws[4]), "+s"(p.ws[5]), "+s"(p.ws[6]), "+s"(p.ws[7]));
  const int rank = p.rank;
  const int rot = bm2;
  char* myws = p.myws;
  const unsigned pd = (unsigned)(b + idx) & 7u;
  const bool early = idx < EARLYN && pd != (unsigned)rank;
  volatile LAS unsigned* ctl = (volatile LAS unsigned*)(size_t)(lds0 + LDS_CTL);

  if (wid < 4) {
    if (wid == 0) TS(0);
    const int wm = wid >> 1, wn = wid & 1;
    const int l15 = lane & 15, g = lane >> 4, qq = l15 >> 2, pp = lane & 3;
    const unsigned aoff = W_TILE + (wm * 32 + l15) * 128 + ((g ^ (l15 >> 1)) << 4);
    const unsigned boff = (8 * g + qq) * 256 + (((wn * 8 + (pp >> 1)) ^ (qq << 2) ^ ((2 * g) & 3)) << 4) + 8 * (pp & 1);
    f32x4 acc[4][4];
#pragma unroll
    for (int m = 0; m < 4; ++m)
#pragma unroll
      for (int n = 0; n < 4; ++n) acc[m][n] = (f32x4){0.f, 0.f, 0.f, 0.f};
    unsigned ra[2][2], rb[2][8];
#pragma unroll
    for (int hh = 0; hh < 2; ++hh) {
#pragma unroll
      for (int k2 = 0; k2 < 2; ++k2) { ra[hh][k2] = lds0 + hh * 65536 + (aoff ^ (k2 << 6)); asm volatile("" : "+v"(ra[hh][k2])); }
#pragma unroll
      for (int u = 0; u < 8; ++u) { rb[hh][u] = lds0 + hh * 65536 + (boff ^ (u << 4)); asm volatile("" : "+v"(rb[hh][u])); }
    }
    auto load = [&](Fr& f, auto slc, auto mc) {
      constexpr int so = decltype(slc)::value * SLOT, hh = so / 65536, o = so % 65536, MT = decltype(mc)::value;
#pragma unroll
      for (int k2 = 0; k2 < 2; ++k2)
#pragma unroll
        for (int n = 0; n < 4; ++n) {
#pragma unroll
          for (int t = 0; t < 2; ++t)
            f.b[k2][n][t] = __builtin_amdgcn_ds_read_tr16_b64_v4bf16((LAS bf16x4*)(size_t)(rb[hh][n * 2 + t] + o + t * 1024 + k2 * 8192));
          if (n < MT) f.a[k2][n] = *(const LAS bf16x8*)(size_t)(ra[hh][k2] + o + (n >> 1) * UNIT + (n & 1) * 2048);
        }
    };
    auto period = [&](Fr& cur, Fr& nxt, auto slc, auto mcur, auto mnxt) {
      constexpr int so = decltype(slc)::value * SLOT, hh = so / 65536, o = so % 65536;
      constexpr int MC = decltype(mcur)::value, MN = decltype(mnxt)::value;
      constexpr int NM = MC * 8, NRD = MN ? 16 + MN * 2 : 0, PER = MC == 4 ? 1 : 2;
      SB();
#pragma unroll
      for (int i = 0; i < NM; ++i) {
        const int n = i / (MC * 2), m = (i >> 1) % MC, k2 = i & 1;
        bf16x8 wv = __builtin_shufflevector(cur.b[k2][n][0], cur.b[k2][n][1], 0, 1, 2, 3, 4, 5, 6, 7);
        MFMA(acc[m][n], wv, cur.a[k2][m]);
#pragma unroll
        for (int rr = i * PER; rr < i * PER + PER; ++rr) {
          if (rr < NRD) {
            if (rr >= 4 && rr < 4 + MN * 2) {
              const int mm = (rr - 4) >> 1, kk = rr & 1;
              nxt.a[kk][mm] = *(const LAS bf16x8*)(size_t)(ra[hh][kk] + o + (mm >> 1) * UNIT + (mm & 1) * 2048);
            } else {
              const int x = rr < 4 ? rr : rr - MN * 2, nn = x >> 2, kk = (x >> 1) & 1, t = x & 1;
              nxt.b[kk][nn][t] = __builtin_amdgcn_ds_read_tr16_b64_v4bf16((LAS bf16x4*)(size_t)(rb[hh][nn * 2 + t] + o + t * 1024 + kk * 8192));
            }
          }
        }
        SB();
      }
    };
    using I0 = std::integral_constant<int, 0>; using I1 = std::integral_constant<int, 1>;
    using I2 = std::integral_constant<int, 2>; using I3 = std::integral_constant<int, 3>; using I4 = std::integral_constant<int, 4>;
#define ENDP() do { WAIT_L0(); BAR(); } while (0)
    Fr f0, f1;
    if (early) BAR();
    BAR();
    if (wid == 0) { TS(3); TC(15); TS(15); }
    load(f0, I0{}, I4{});
    const unsigned call = __builtin_amdgcn_readfirstlane(ctl[64]) + 1;
    ENDP();
    constexpr int NPL = (NP1 - 1) / 4 * 4;
    for (int pr = 0; pr < NPL; pr += 4) {
      period(f0, f1, I1{}, I4{}, I4{}); ENDP();
      period(f1, f0, I2{}, I4{}, I4{}); ENDP();
      period(f0, f1, I3{}, I4{}, I4{}); ENDP();
      period(f1, f0, I0{}, I4{}, I4{}); ENDP();
    }
    const unsigned tagh = call << 16;
    const char* xsrc = myws + WS_XCH + (size_t)(b * 4 + wid) * XB + 4096;
    const unsigned xl = lds0 + LDS_X + wid * XB;
    auto xfetch = [&]() {
      asm volatile("s_mov_b32 m0, %2\n\ts_nop 0\n\t" GL_(-4096) GL_(-3072) GL_(-2048) GL_(-1024) GL_(0) GL_(1024) ::"v"(lane * 16), "s"(xsrc), "s"(xl + 4096) : "memory");
    };
    auto tail = [&](auto self, auto pc) {
      constexpr int P = decltype(pc)::value;
      if constexpr (P == NP1) {
        asm volatile("s_nop 7" : "+v"(acc[3][3]), "+v"(acc[2][3]), "+v"(acc[3][2]));
        if (wid == 0) { TS(6); TC(22); TS(22); }
        char* xd = myws + WS_XCH + (size_t)((b ^ 4) * 4 + wid) * XB + lane * 16;
        auto val = [&](int i) { return i < 32 ? acc[2 + (i >> 4)][(i >> 2) & 3][i & 3] : 0.f; };
#pragma unroll
        for (int i = 0; i < 6; ++i) {
          const unsigned t = pk(val(6 * i + 2), val(6 * i + 5));
          st16_sys(xd + i * 1024, (u32x4){pk(val(6 * i), val(6 * i + 1)), (t & 0xffffu) | tagh, pk(val(6 * i + 3), val(6 * i + 4)), (t >> 16) | tagh});
        }
      }
      if constexpr (P == NP - XAT) xfetch();
      using SLc = std::integral_constant<int, (P + 1) & 3>;
      using MCc = std::integral_constant<int, (P < NP1) ? 4 : 2>;
      using MNc = std::integral_constant<int, (P + 1 < NP1) ? 4 : (P + 1 < NP) ? 2 : 0>;
      if constexpr (P & 1) period(f1, f0, SLc{}, MCc{}, MNc{}); else period(f0, f1, SLc{}, MCc{}, MNc{});
      if constexpr (P + 2 < NP) ENDP();
      if constexpr (P + 1 < NP) self(self, std::integral_constant<int, P + 1>{});
    };
    tail(tail, std::integral_constant<int, NPL>{});
    bool fail = __builtin_amdgcn_readfirstlane(ctl[0]) != 0;
    u32x4 xp[6];
    {
      u64 t0 = 0;
      while (true) {
        WAIT_V(0);
        unsigned bad = 0;
#pragma unroll
        for (int i = 0; i < 6; ++i) { xp[i] = *(volatile LAS u32x4*)(size_t)(xl + i * 1024 + lane * 16); bad |= (xp[i][1] ^ tagh) | (xp[i][3] ^ tagh); }
        if (__all(bad < 0x10000u) || fail) break;
        const u64 now = __builtin_amdgcn_s_memrealtime();
        if (t0 == 0) t0 = now;
        if (now - t0 > DEADLINE) fail = true;
        xfetch();
      }
    }
    asm volatile("s_nop 7\n\ts_nop 1" : "+v"(acc[1][3]), "+v"(acc[0][3]), "+v"(acc[1][2]));
    if (wid == 0) { TS(1); TC(23); TS(23); }
    char* obase = p.out + ((size_t)((bm2 * 2 + q) * 64 + wm * 32 + l15) * NN + bn * 128 + wn * 64) * 2 + ((g & 1) * 16 + (g >> 1) * 8) * 2;
#pragma unroll
    for (int m = 0; m < 2; ++m)
#pragma unroll
      for (int n = 0; n < 4; n += 2) {
        const f32x4 c0 = acc[m][n], c1 = acc[m][n + 1];
        auto xv = [&](int i) {
          const int r6 = i % 6, w = r6 < 2 ? 0 : r6 == 2 ? 1 : r6 < 5 ? 2 : 3;
          const unsigned v = xp[i / 6][w];
          return __uint_as_float(r6 == 1 || r6 == 4 ? v & 0xffff0000u : v << 16);
        };
        const int i0 = m * 16 + n * 4;
        const unsigned a0 = pk(c0[0] + xv(i0), c0[1] + xv(i0 + 1)), a1 = pk(c0[2] + xv(i0 + 2), c0[3] + xv(i0 + 3));
        const unsigned b0 = pk(c1[0] + xv(i0 + 4), c1[1] + xv(i0 + 5)), b1 = pk(c1[2] + xv(i0 + 6), c1[3] + xv(i0 + 7));
        auto r0 = __builtin_amdgcn_permlane16_swap(a0, b0, false, false);
        auto r1 = __builtin_amdgcn_permlane16_swap(a1, b1, false, false);
        u32x4 v = {r0[0], r1[0], r0[1], r1[1]};
        if (fail) v = (u32x4){0x7fc07fc0u, 0x7fc07fc0u, 0x7fc07fc0u, 0x7fc07fc0u};
        st16_sys(obase + (size_t)m * 16 * NN * 2 + n * 32, v);
      }
  } else if (wid == 4) {
    const int r = lane >> 4, c = lane & 15;
    unsigned voff[8];
#pragma unroll
    for (int u = 0; u < 8; ++u) voff[u] = (4 * u + r) * (NN * 2) + ((c ^ (r << 2) ^ (u & 3)) << 4) + 4096 - u * 1024;
    const char* base = p.w + bn * 256;
    auto issue = [&](const char* sb, unsigned la) { dma8k_v(sb, voff, la); dma8k_v(sb + 32 * NN * 2, voff, la + 8192); };
    auto src = [&](int code) { return base + (size_t)code * (64 * NN * 2); };
    if (early) BAR(); else __builtin_amdgcn_s_sleep(HOLD);
    issue(src(rank * 8 + 2 * (rot & 3) + q), lds0 + 4096);
    issue(src(rank * 8 + 2 * ((1 + rot) & 3) + q), lds0 + 4096 + SLOT);
    const int tab = sched_code(lane < NP ? lane : 0, rank, q, rot, bn & 3);
    WAIT_V(16);
    BAR();
    {
      const char* s4 = src(__builtin_amdgcn_readlane(tab, 4));
      const unsigned po = (lane >> 1) * (NN * 2) + (lane & 1) * 128;
      dma4(s4, po, lds0 + LDS_JNK); dma4(s4 + 32 * NN * 2, po, lds0 + LDS_JNK);
      WAIT_V(2);
    }
    BAR();
    issue(src(rank * 8 + 2 * ((2 + rot) & 3) + q), lds0 + 4096 + 2 * SLOT);
    issue(src(rank * 8 + 2 * ((3 + rot) & 3) + q), lds0 + 4096 + 3 * SLOT);
    WAIT_V(16);
    BAR();
    issue(src(__builtin_amdgcn_readlane(tab, 4)), lds0 + 4096);
    const char* sb = src(__builtin_amdgcn_readlane(tab, 5));
    unsigned la = lds0 + 4096 + SLOT;
    for (int k = 2; k < NP - D; ++k) {
      dma8k_v(sb, voff, la);
      const char* sb2 = sb + 32 * NN * 2;
      unsigned la2 = la + 8192;
      sb = src(__builtin_amdgcn_readlane(tab, k + D + 1));
      la = lds0 + 4096 + ((k + D + 1) & 3) * SLOT;
      asm volatile("" : "+s"(sb), "+s"(la), "+s"(sb2), "+s"(la2));
      dma8k_v(sb2, voff, la2);
      WAIT_V(32);
      BAR();
    }
    WAIT_V(16); BAR();
    WAIT_V(0); BAR();
  } else {
    const int r = lane >> 3, c = lane & 7;
    unsigned voffx[8];
#pragma unroll
    for (int u = 0; u < 8; ++u) voffx[u] = (8 * u + r) * (KLOC * 2) + ((c ^ (((u & 1) << 2) | (r >> 1))) << 4) + 4096 - u * 1024;
    char* cnt = myws + WS_CNT + b * 128;
    volatile LAS unsigned* fl = (volatile LAS unsigned*)(size_t)(lds0 + LDS_FLG + lane * 4);
    if (wid == 5) {
      const u64 t_start = __builtin_amdgcn_s_memrealtime();
      const char* xb = p.x + (size_t)(rank * MLOC + bm2 * 128) * (KLOC * 2);
      auto issue_local = [&](int e) {
        const char* sb = xb + (2 * ((e + rot) & 3) + q) * 128;
        const unsigned la = lds0 + e * SLOT + W_TILE + 4096;
        dma8k_v(sb + q * (64 * KLOC * 2), voffx, la);
        dma8k_v(sb + (q ^ 1) * (64 * KLOC * 2), voffx, la + UNIT);
      };
      fl[0] = 0; ctl[0] = 0;
      if (early) BAR(); else __builtin_amdgcn_s_sleep(HOLD);
      dma4_sys(cnt, 0u, lds0 + LDS_CNT);
      issue_local(0); issue_local(1);
      const int tab = sched_code(lane < NP ? lane : 0, rank, q, rot, bn & 3);
      WAIT_V(16);
      const unsigned call = __builtin_amdgcn_readfirstlane(ctl[64]) + 1;
      const char* db = myws + (call & 1) * WS_BUF + (size_t)(bm2 * 2) * 8 * UNIT + 4096;
      const char* fp = myws + WS_FLAG + bm2 * 8192;
      const unsigned vlin = lane * 16;
      const bool nocheck = lane < 4 || lane >= NP;
      const unsigned fvo = tab * 128;
      const unsigned texp = PUSH0 + VIS + slot_time((tab & 7) * 4 + ((((((tab >> 3) - rank - 1) & 7) >> 1) - bm2) & 3));
      const unsigned aoffs = (tab >> 3) * (MLOC * KLOC * 2) + (tab & 7) * UNIT;
      int avail = 4;
      int since = 0;
      const unsigned fthr = nocheck ? 0u : call;
      const unsigned tlow = (unsigned)t_start;
      auto poll_fire = [&](int upto, unsigned now) {
        const u64 m = __ballot((lane >= avail) & (lane < NP) & ((lane <= upto) | (texp <= now - tlow)));
        asm volatile("s_mov_b32 m0, %2\n\ts_mov_b64 exec, %3\n\tglobal_load_lds_dword %0, %1 sc0 sc1\n\ts_mov_b64 exec, -1" ::"v"(fvo), "s"(fp), "s"(lds0 + LDS_FLG), "s"(m) : "memory");
        ++since;
      };
      auto poll_issue = [&](int upto) { poll_fire(upto, (unsigned)__builtin_amdgcn_s_memrealtime()); };
      auto poll_read = [&]() { avail = first_clear(*fl >= fthr); };
      WAIT_L0();
      BAR();
      st4_sys_l0(cnt, call);
      WAIT_V(1);
      BAR();
      issue_local(2);
      issue_local(3);
      poll_issue(FP0);
      WAIT_V(17);
      BAR();
      WAIT_V(0);
      poll_read();
      int na = 4;
      auto issue = [&]() {
        const char* sb = db + (unsigned)__builtin_amdgcn_readlane(aoffs, na);
        const unsigned la = lds0 + (na & 3) * SLOT + W_TILE + 4096;
        dma8k_lin(sb + q * (8 * UNIT), vlin, la);
        if (na < NP1) dma8k_lin(sb + (q ^ 1) * (8 * UNIT), vlin, la + UNIT);
        ++na; since = 0;
      };
      auto gate = [&](int k) {
        if (avail < k + DA + 1 + PTH && avail < NP) {
          poll_read();
          if (avail <= k + DA && avail > k + 1) {
            poll_issue(k + DA + RUP);
            WAIT_V(0);
            poll_read();
          }
          if (avail <= k + 1) {
            const u64 t0 = __builtin_amdgcn_s_memrealtime();
            while (true) {
              poll_issue(k + 2);
              WAIT_V(0);
              poll_read();
              if (avail > k + 1) break;
              if (__builtin_amdgcn_s_memrealtime() - t0 > DEADLINE) { avail = 64; *fl = ~0u; ctl[0] = 1; WAIT_L0(); break; }
              __builtin_amdgcn_s_sleep(1);
            }
          }
          if (avail < k + DA + 1 + PTH && avail < NP) poll_issue(avail);
        }
      };
      int kpoll = 0;
      const char* db0 = db + q * (8 * UNIT);
      const char* db1 = db + (q ^ 1) * (8 * UNIT);
      for (int k = 2; k < NP1 - DA; ++k) {
        if (na == k + DA && avail > na) {
          const unsigned o = __builtin_amdgcn_readlane(aoffs, na);
          const unsigned la = lds0 + (na & 3) * SLOT + W_TILE + 4096;
          const char* s0 = db0 + o; const char* s1 = db1 + o;
          unsigned la1 = la + UNIT;
          ++na;
          asm volatile("" : "+s"(s1), "+s"(la1), "+s"(na));
          if (k < kpoll) {
            dma8k_lin(s0, vlin, la);
            SB();
            const unsigned fv = *fl;
            SB();
            dma8k_lin(s1, vlin, la1);
            WAIT_V(16 * (DA - 1));
            BAR();
            avail = first_clear(fv >= fthr);
          } else if (avail < k + DA + 1 + PTH) {
            const unsigned now = (unsigned)__builtin_amdgcn_s_memrealtime();
            SB();
            dma8k_lin(s0, vlin, la);
            SB();
            poll_fire(avail, now); kpoll = k + 2;
            SB();
            dma8k_lin(s1, vlin, la1);
            WAIT_V(16 * (DA - 1) + 1);
            BAR();
          } else {
            dma8k_lin(s0, vlin, la);
            dma8k_lin(s1, vlin, la1);
            WAIT_V(16 * (DA - 1));
            BAR();
          }
          since = 0;
          continue;
        } else {
          gate(k);
          kpoll = k + 2;
          const int lim = k + DA + 1;
          const int top = avail < lim ? avail : lim;
          while (na < top) issue();
          const int ahead = na - 2 - k;
          if (ahead >= 2) WAIT_V(32); else if (ahead == 1) WAIT_V(16);
          else if (since >= 3) WAIT_V(3); else if (since == 2) WAIT_V(2); else if (since == 1) WAIT_V(1); else WAIT_V(0);
        }
        BAR();
      }
      for (int k = NP1 - DA; k < NP - 1; ++k) {
        const int lim = k + D + 1 > NP ? NP : k + D + 1;
        gate(k);
        const int top = avail < lim ? avail : lim;
        while (na < top) issue();
        const int ahead = na - 2 - k;
        if (ahead >= 2) WAIT_V(16); else if (ahead == 1) WAIT_V(8); else WAIT_V(0);
        BAR();
      }
    } else {
      const u64 t_start = __builtin_amdgcn_s_memrealtime();
      const int pj = idx >> 2;
      const int prb = ((((rank - (int)pd - 1) & 7) >> 1) - idx) & 3;
      if (pd == (unsigned)rank) return;
      const char* ps = p.x + (size_t)(pd * MLOC + prb * 128) * (KLOC * 2) + pj * 128;
      const unsigned stg = lds0 + LDS_X;
      auto stage = [&]() { dma8k_v(ps, voffx, stg + 4096); dma8k_v(ps + 64 * KLOC * 2, voffx, stg + UNIT + 4096); };
      u32x4 pv[16];
      unsigned call, cv;
      if (pj <= 1) {
        STAMP(st1);
        ld8k(ps, voffx, pv);
        asm volatile("global_load_dword %0, %1, %2 sc0 sc1" : "=v"(cv) : "v"(0u), "s"(cnt) : "memory");
        ld8k(ps + 64 * KLOC * 2, voffx, pv + 8);
        STAMP(st2);
        SB();
        char* pw = p.ws[0];
#pragma unroll
        for (int i = 1; i < NR; ++i) pw = pd == (unsigned)i ? p.ws[i] : pw;
        const char* pe0 = pw + (size_t)((rank * 8 + prb * 2) * 8 + pj) * UNIT + 4096;
        unsigned vl = lane * 16;
        asm volatile("" : "+s"(pe0));
        asm volatile("" : "+v"(vl));
        asm volatile("s_waitcnt vmcnt(0)" : "+v"(cv), "+v"(pv[0]), "+v"(pv[1]), "+v"(pv[2]), "+v"(pv[3]), "+v"(pv[4]), "+v"(pv[5]), "+v"(pv[6]), "+v"(pv[7]),
                     "+v"(pv[8]), "+v"(pv[9]), "+v"(pv[10]), "+v"(pv[11]), "+v"(pv[12]), "+v"(pv[13]), "+v"(pv[14]), "+v"(pv[15]) :: "memory");
        STAMP(st3);
        if (idx == 0) {
          const unsigned vo = vl + ((~cv & 1u) << 22);
          st8k_sys(pe0, vo, pv);
          st8k_sys(pe0 + 8 * UNIT, vo, pv + 8);
        }
        call = __builtin_amdgcn_readfirstlane(cv) + 1;
      } else {
        BAR();
        call = __builtin_amdgcn_readfirstlane(ctl[64]) + 1;
      }
      char* pw = p.ws[0];
#pragma unroll
      for (int i = 1; i < NR; ++i) pw = pd == (unsigned)i ? p.ws[i] : pw;
      const u64 t_push = t_start + PUSH0 + slot_time(idx);
      const char* pd0 = pw + (call & 1) * WS_BUF + (size_t)((rank * 8 + prb * 2) * 8 + pj) * UNIT + 4096;
      char* pflag = pw + WS_FLAG + ((prb * 8 + rank) * 8 + pj) * 128;
      bool starving = true;
      if (pj != 0) {
        const int tab = sched_code(lane < NP ? lane : 0, rank, q, rot, bn & 3);
        int k = pj == 1 ? -1 : 0;
        auto run_until = [&](u64 t) {
          for (;; ++k) {
            starving = true;
            const int code = __builtin_amdgcn_readlane(tab, k + 1);
            if (k >= NP1 - 2 || (k >= 3 && (code & 7) >= pj)) break;
            bool go = false;
            while (true) {
              starving = k >= 3 && (unsigned)__builtin_amdgcn_readlane(*fl, k + 1) < call;
              if (__builtin_amdgcn_s_memrealtime() + (starving ? 0 : k < 5 ? PE1 : PEAR) >= t) { go = true; break; }
              if (!starving) break;
              __builtin_amdgcn_s_sleep(2);
            }
            if (go) break;
            BAR();
          }
        };
        if (pj >= 2) {
          const int kst = pj < PJL ? KST : pj < PJM ? 5 - q : 8;
          for (; k < kst; ++k) BAR();
          __builtin_amdgcn_s_sleep(KSL);
          stage();
          BAR(); ++k;
          if (kst > 3) { BAR(); ++k; }
          WAIT_V(0);
#pragma unroll
          for (int i = 0; i < 16; ++i) pv[i] = *(volatile LAS u32x4*)(size_t)(stg + i * 1024 + lane * 16);
        }
        run_until(t_push);
        while (k < 5 && !starving && __builtin_amdgcn_s_memrealtime() < t_push) __builtin_amdgcn_s_sleep(1);
      } else {
        while (idx != 0 && __builtin_amdgcn_s_memrealtime() < t_push) __builtin_amdgcn_s_sleep(1);
      }
      STAMP(st4);
      if (idx != 0) {
        st8k_sys(pd0, lane * 16, pv);
        st8k_sys(pd0 + 8 * UNIT, lane * 16, pv + 8);
      }
      TS(5);
      if (!starving || early) BAR();
      WAIT_V(0);
      asm volatile("s_mov_b64 exec, 0xffffffff\n\tglobal_store_dword %0, %1, off sc0 sc1\n\ts_mov_b64 exec, -1" ::"v"(pflag + lane * 4), "v"(call) : "memory");
      TS(7);
      BAR(); BAR();
    }
  }
}
}

extern "C" void kernel_launch(int rank, int nranks, ncclComm_t comm, hipStream_t stream,
                              void* const* d_in, const int* in_sizes, int n_in,
                              void* d_out, int out_size, const int* in_dtypes, int out_dtype,
                              void* const* d_out_all, void* const* d_ws_all, size_t ws_bytes) {
  if (nranks != NR || ws_bytes < WS_END || (unsigned)rank >= (unsigned)NR) { fprintf(stderr, "bad config\n"); return; }
  static bool init = false;
  if (!init) {
    HIP_CHECK(hipFuncSetAttribute((const void*)a2a_gemm, hipFuncAttributeMaxDynamicSharedMemorySize, LDS_BYTES));
    init = true;
  }
  Params p;
  p.x = (const char*)d_in[0];
  p.w = (const char*)d_in[1];
  p.out = (char*)d_out;
  for (int i = 0; i < NR; ++i) p.ws[i] = (char*)d_ws_all[i];
  p.myws = (char*)d_ws_all[rank];
  p.rank = rank;
  p.pad = 0;
  hipLaunchKernelGGL(a2a_gemm, dim3(256), dim3(448), LDS_BYTES, stream, p);
  HIP_CHECK(hipGetLastError());
}
```

```cpp
#include <hip/hip_runtime.h>
#include <rccl/rccl.h>
#include <cstdio>
#include <type_traits>

#define HIP_CHECK(expr) do { hipError_t _e = (expr); if (_e != hipSuccess) { \
    fprintf(stderr, "HIP error %s at %s:%d\n", hipGetErrorString(_e), __FILE__, __LINE__); \
    return; } } while (0)

typedef unsigned u32x4 __attribute__((ext_vector_type(4)));
typedef float f32x4 __attribute__((ext_vector_type(4)));
typedef __bf16 bf16x4 __attribute__((ext_vector_type(4)));
typedef __bf16 bf16x8 __attribute__((ext_vector_type(8)));
typedef __bf16 bf16x2 __attribute__((ext_vector_type(2)));
typedef unsigned long long u64;
#define LAS __attribute__((address_space(3)))

namespace {
constexpr int NR = 8, MLOC = 512, KLOC = 512, NN = 4096;
constexpr int W_TILE = 128 * 64 * 2;
constexpr int UNIT = 64 * 64 * 2;
constexpr int SLOT = W_TILE + 2 * UNIT;
constexpr int S = 4, D = 3;
#define EB 2
#define XPP 5
#define XOFF 10
#define XGAP 1
#define PJR 1
#define KNB -9
#define SKP0 99
#define SKP1 99
#define FSTEPS 8
constexpr int F = FSTEPS;
constexpr int NP1 = (64 - F) / 2;
constexpr int NP = NP1 + F;
static_assert(F % 2 == 0 && F >= 2 && F <= 16, "phase 2 length");
#define XAT 3
constexpr int XB = 5 * 1024;
constexpr int LDS_X = S * SLOT;
constexpr int LDS_CTL = LDS_X + 4 * XB;
constexpr int LDS_CNT = LDS_CTL + 256;
constexpr int LDS_FLG = LDS_CNT + 512;
constexpr int LDS_JNK = LDS_FLG + 256;
constexpr int LDS_BYTES = LDS_JNK + 256;
#define TAU 148
#define PTH 3
#define EARLYN 3
#define KST 0
#define PJL 9
#define PJM 6
#define KSL 0
#define RUP 1
#define FP0 5
#define DA 2
#define VIS 200
#define TAU1 132
#define NS1 8
#define PE1 50
__device__ __forceinline__ int slot_time(int s) { return s < NS1 ? (s * TAU1 >> 2) : ((NS1 * TAU1 + (s - NS1) * TAU) >> 2); }
#define PEAR 20
#define HOLD 12
#define PUSH0 58
constexpr size_t WS_BUF = (size_t)NR * MLOC * KLOC * 2;
constexpr size_t WS_XCH = 2 * WS_BUF;
constexpr size_t WS_FLAG = WS_XCH + 256 * 4 * XB;
constexpr size_t WS_CNT = WS_FLAG + 256 * 128;
constexpr size_t WS_END = WS_CNT + 256 * 128;
constexpr u64 DEADLINE = 50000000ull;

struct Params {
  const char* x;
  const char* w;
  char* out;
  char* myws;
  int rank;
  int pad;
  char* ws[NR];
};

#define WAIT_V(n) asm volatile("s_waitcnt vmcnt(%0)" ::"n"(n) : "memory")
#define WAIT_L0() __builtin_amdgcn_s_waitcnt(0xc07f)
#define BAR() __builtin_amdgcn_s_barrier()
#define SB() __builtin_amdgcn_sched_barrier(0)

__device__ __forceinline__ void st16_sys(void* p, u32x4 v) {
#define SPOL " sc0 sc1"
  asm volatile("global_store_dwordx4 %0, %1, off" SPOL "\n\ts_nop 1" ::"v"(p), "v"(v) : "memory");
}
__device__ __forceinline__ void st16_sys_s(const char* sbase, unsigned voff, u32x4 v) {
  asm volatile("global_store_dwordx4 %0, %1, %2 sc0 sc1\n\ts_nop 1" ::"v"(voff), "v"(v), "s"(sbase) : "memory");
}
__device__ __forceinline__ void st4_sys_l0(void* p, unsigned v) {
  asm volatile("s_mov_b64 exec, 1\n\tglobal_store_dword %0, %1, off sc0 sc1\n\ts_mov_b64 exec, -1" ::"v"(p), "v"(v) : "memory");
}
__device__ __forceinline__ unsigned ld4_sys(const void* p) {
  unsigned v;
  asm volatile("global_load_dword %0, %1, off sc0 sc1\n\ts_waitcnt vmcnt(0)" : "=&v"(v) : "v"(p) : "memory");
  return v;
}
#define APOL " sc0 sc1"
#define GL_(o) "global_load_lds_dwordx4 %0, %1 offset:" #o APOL "\n\t"
__device__ __forceinline__ void dma8k_lin(const char* sbase, unsigned voff, unsigned ldsaddr) {
  asm volatile("s_mov_b32 m0, %2\n\ts_nop 0\n\t" GL_(-4096) GL_(-3072) GL_(-2048) GL_(-1024) GL_(0) GL_(1024) GL_(2048) GL_(3072)
               ::"v"(voff), "s"(sbase), "s"(ldsaddr) : "memory");
}
#define WPOL ""
#define GV_(i, o) "global_load_lds_dwordx4 %" #i ", %8 offset:" #o WPOL "\n\t"
__device__ __forceinline__ void dma8k_v(const char* sbase, const unsigned (&v)[8], unsigned ldsaddr) {
  asm volatile("s_mov_b32 m0, %9\n\ts_nop 0\n\t" GV_(0, -4096) GV_(1, -3072) GV_(2, -2048) GV_(3, -1024) GV_(4, 0) GV_(5, 1024) GV_(6, 2048) GV_(7, 3072)
               ::"v"(v[0]), "v"(v[1]), "v"(v[2]), "v"(v[3]), "v"(v[4]), "v"(v[5]), "v"(v[6]), "v"(v[7]), "s"(sbase), "s"(ldsaddr) : "memory");
}
#define GS_(i, o) "global_store_dwordx4 %8, %" #i ", %9 offset:" #o " sc0 sc1\n\t"
__device__ __forceinline__ void st8k_sys(const char* sbase, unsigned voff, const u32x4* v) {
  asm volatile(GS_(0, -4096) GS_(1, -3072) GS_(2, -2048) GS_(3, -1024) GS_(4, 0) GS_(5, 1024) GS_(6, 2048) GS_(7, 3072) "s_nop 1"
               ::"v"(v[0]), "v"(v[1]), "v"(v[2]), "v"(v[3]), "v"(v[4]), "v"(v[5]), "v"(v[6]), "v"(v[7]), "v"(voff), "s"(sbase) : "memory");
}
#define GR_(i, a, o) "global_load_dwordx4 %" #i ", %" #a ", %16 offset:" #o "\n\t"
__device__ __forceinline__ void ld8k(const char* sbase, const unsigned (&v)[8], u32x4* d) {
  asm volatile(GR_(0, 8, -4096) GR_(1, 9, -3072) GR_(2, 10, -2048) GR_(3, 11, -1024) GR_(4, 12, 0) GR_(5, 13, 1024) GR_(6, 14, 2048) GR_(7, 15, 3072)
               : "=&v"(d[0]), "=&v"(d[1]), "=&v"(d[2]), "=&v"(d[3]), "=&v"(d[4]), "=&v"(d[5]), "=&v"(d[6]), "=&v"(d[7])
               : "v"(v[0]), "v"(v[1]), "v"(v[2]), "v"(v[3]), "v"(v[4]), "v"(v[5]), "v"(v[6]), "v"(v[7]), "s"(sbase) : "memory");
}
#define MFMA(c, x, y) asm volatile("v_mfma_f32_16x16x32_bf16 %0, %1, %2, %0" : "+v"(c) : "v"(x), "v"(y))
__device__ __forceinline__ void dma4_sys(const char* sbase, unsigned voff, unsigned ldsaddr) {
  asm volatile("s_mov_b32 m0, %2\n\ts_nop 0\n\tglobal_load_lds_dword %0, %1 sc0 sc1" ::"v"(voff), "s"(sbase), "s"(ldsaddr) : "memory");
}
__device__ __forceinline__ void dma4(const char* sbase, unsigned voff, unsigned ldsaddr) {
  asm volatile("s_mov_b32 m0, %2\n\ts_nop 0\n\tglobal_load_lds_dword %0, %1" ::"v"(voff), "s"(sbase), "s"(ldsaddr) : "memory");
}
__device__ __forceinline__ int first_clear(bool c) {
  const u64 m = ~__ballot(c);
  return __builtin_amdgcn_readfirstlane(m ? __builtin_ctzll(m) : 64);
}
__device__ __forceinline__ unsigned pk(float a, float b) { return __builtin_bit_cast(unsigned, (bf16x2){(__bf16)a, (__bf16)b}); }

__device__ __forceinline__ int sched_code(int e, int rank, int q, int rot, int phi) {
  if (e < 4) return rank * 8 + 2 * ((e + rot) & 3) + q;
  int u;
  if (e < NP1) {
    const int u0 = 2 * (e - 4) + q;
    const int r = u0 / 7;
    const int first = 7 * r + ((7 * r + q) & 1);
    int last = 7 * r + 6; if (last > 55 - F) last = 55 - F;
    const int c = (last - first) / 2 + 1;
    u = first + 2 * (((u0 - first) / 2 + rot) % c);
  } else {
    u = 56 - F + ((e - NP1 + rot * 2 + q) % F);
  }
  const int r = u / 7, i = u - 7 * r;
  return ((rank + 1 + i) & 7) * 8 + r;
}

#define TC(i) do {} while (0)
#define TS(i) do {} while (0)

struct Fr { bf16x8 a[2][4]; bf16x4 b[2][4][2]; };

__global__ __launch_bounds__(448, 2) void a2a_gemm(Params p) {
  extern __shared__ __attribute__((aligned(1024))) char smem[];
#define STAMP(x) do {} while (0)
  const unsigned lds0 = (unsigned)(size_t)(LAS char*)smem;
  const int b = blockIdx.x;
  const int wid = __builtin_amdgcn_readfirstlane(threadIdx.x >> 6);
  const int lane = threadIdx.x & 63;
  const int xcd = b & 7, idx = b >> 3;
  const int q = xcd >> 2, bn = (xcd & 3) * 8 + (idx & 7), bm2 = idx >> 3;
  asm volatile("" : "+s"(p.x), "+s"(p.w), "+s"(p.out), "+s"(p.myws), "+s"(p.rank), "+s"(p.ws[0]), "+s"(p.ws[1]), "+s"(p.ws[2]), "+s"(p.ws[3]),
               "+s"(p.ws[4]), "+s"(p.ws[5]), "+s"(p.ws[6]), "+s"(p.ws[7]));
  const int rank = p.rank;
  const int rot = bm2;
  char* myws = p.myws;
  const unsigned pd = (unsigned)(b + idx) & 7u;
  const bool early = idx < EARLYN && pd != (unsigned)rank;
  volatile LAS unsigned* ctl = (volatile LAS unsigned*)(size_t)(lds0 + LDS_CTL);

  if (wid < 4) {
    if (wid == 0) TS(0);
    const int wm = wid >> 1, wn = wid & 1;
    const int l15 = lane & 15, g = lane >> 4, qq = l15 >> 2, pp = lane & 3;
    const unsigned aoff = W_TILE + (wm * 32 + l15) * 128 + ((g ^ (l15 >> 1)) << 4);
    const unsigned boff = (8 * g + qq) * 256 + (((wn * 8 + (pp >> 1)) ^ (qq << 2) ^ ((2 * g) & 3)) << 4) + 8 * (pp & 1);
    f32x4 acc[4][4];
#pragma unroll
    for (int m = 0; m < 4; ++m)
#pragma unroll
      for (int n = 0; n < 4; ++n) acc[m][n] = (f32x4){0.f, 0.f, 0.f, 0.f};
    unsigned ra[2][2], rb[2][8];
#pragma unroll
    for (int hh = 0; hh < 2; ++hh) {
#pragma unroll
      for (int k2 = 0; k2 < 2; ++k2) { ra[hh][k2] = lds0 + hh * 65536 + (aoff ^ (k2 << 6)); asm volatile("" : "+v"(ra[hh][k2])); }
#pragma unroll
      for (int u = 0; u < 8; ++u) { rb[hh][u] = lds0 + hh * 65536 + (boff ^ (u << 4)); asm volatile("" : "+v"(rb[hh][u])); }
    }
    auto load = [&](Fr& f, auto slc, auto mc) {
      constexpr int so = decltype(slc)::value * SLOT, hh = so / 65536, o = so % 65536, MT = decltype(mc)::value;
#pragma unroll
      for (int k2 = 0; k2 < 2; ++k2)
#pragma unroll
        for (int n = 0; n < 4; ++n) {
#pragma unroll
          for (int t = 0; t < 2; ++t)
            f.b[k2][n][t] = __builtin_amdgcn_ds_read_tr16_b64_v4bf16((LAS bf16x4*)(size_t)(rb[hh][n * 2 + t] + o + t * 1024 + k2 * 8192));
          if (n < MT) f.a[k2][n] = *(const LAS bf16x8*)(size_t)(ra[hh][k2] + o + (n >> 1) * UNIT + (n & 1) * 2048);
        }
    };
    auto period = [&](Fr& cur, Fr& nxt, auto slc, auto mcur, auto mnxt, auto hook) {
      constexpr int so = decltype(slc)::value * SLOT, hh = so / 65536, o = so % 65536;
      constexpr int MC = decltype(mcur)::value, MN = decltype(mnxt)::value;
      constexpr int NM = MC * 8, NRD = MN ? 16 + MN * 2 : 0, PER = MC == 4 ? 1 : 2;
      constexpr bool EBAR = MC == 4;
      SB();
#pragma unroll
      for (int i = 0; i < NM; ++i) {
        const int n = i / (MC * 2), m = (i >> 1) % MC, k2 = i & 1;
        bf16x8 wv = __builtin_shufflevector(cur.b[k2][n][0], cur.b[k2][n][1], 0, 1, 2, 3, 4, 5, 6, 7);
        MFMA(acc[m][n], wv, cur.a[k2][m]);
        int r0 = i * PER, r1 = r0 + PER;
#pragma unroll
        for (int rr = r0; rr < r1; ++rr) {
          if (rr < NRD) {
            if (rr >= 4 && rr < 4 + MN * 2) {
              const int mm = (rr - 4) >> 1, kk = rr & 1;
              nxt.a[kk][mm] = *(const LAS bf16x8*)(size_t)(ra[hh][kk] + o + (mm >> 1) * UNIT + (mm & 1) * 2048);
            } else {
              const int x = rr < 4 ? rr : rr - MN * 2, nn = x >> 2, kk = (x >> 1) & 1, t = x & 1;
              nxt.b[kk][nn][t] = __builtin_amdgcn_ds_read_tr16_b64_v4bf16((LAS bf16x4*)(size_t)(rb[hh][nn * 2 + t] + o + t * 1024 + kk * 8192));
            }
          }
        }
        hook(i);
        if (EBAR && i == NM - 1 - EB) { SB(); WAIT_L0(); BAR(); }
        SB();
      }
    };
    using I0 = std::integral_constant<int, 0>; using I1 = std::integral_constant<int, 1>;
    using I2 = std::integral_constant<int, 2>; using I3 = std::integral_constant<int, 3>; using I4 = std::integral_constant<int, 4>;
#define ENDP() do { WAIT_L0(); BAR(); } while (0)
#define ENDP1() do {} while (0)
    auto nohook = [](int) {};
    Fr f0, f1;
    if (early) BAR();
    BAR();
    if (wid == 0) { TS(3); TC(15); TS(15); }
    load(f0, I0{}, I4{});
    const unsigned call = __builtin_amdgcn_readfirstlane(ctl[64]) + 1;
    ENDP();
    constexpr int NPL = (NP1 - 1) / 4 * 4;
    for (int pr = 0; pr < NPL; pr += 4) {
      period(f0, f1, I1{}, I4{}, I4{}, nohook); ENDP1();
      period(f1, f0, I2{}, I4{}, I4{}, nohook); ENDP1();
      period(f0, f1, I3{}, I4{}, I4{}, nohook); ENDP1();
      period(f1, f0, I0{}, I4{}, I4{}, nohook); ENDP1();
    }
    const unsigned tagh = call << 16;
    const char* xsrc = myws + WS_XCH + (size_t)(b * 4 + wid) * XB + 4096;
    const char* xdst = myws + WS_XCH + (size_t)((b ^ 4) * 4 + wid) * XB;
    const unsigned xl = lds0 + LDS_X + wid * XB;
    auto xfetch = [&]() {
      asm volatile("s_mov_b32 m0, %2\n\ts_nop 0\n\t" GL_(-4096) GL_(-3072) GL_(-2048) GL_(-1024) GL_(0) ::"v"(lane * 16), "s"(xsrc), "s"(xl + 4096) : "memory");
    };
    auto xsend = [&](int i) {
      auto val = [&](int v) { return v < 32 ? acc[2 + (v >> 4)][(v >> 2) & 3][v & 3] : 0.f; };
      const u32x4 d = {pk(val(7 * i), val(7 * i + 1)), pk(val(7 * i + 2), val(7 * i + 3)), pk(val(7 * i + 4), val(7 * i + 5)), (pk(val(7 * i + 6), 0.f) & 0xffffu) | tagh};
      st16_sys_s(xdst + i * 1024, lane * 16, d);
    };
    u32x4 xp[5];
    unsigned failw = 0;
    auto xread = [&]() {
      WAIT_V(0);
#pragma unroll
      for (int i = 0; i < 5; ++i) xp[i] = *(volatile LAS u32x4*)(size_t)(xl + i * 1024 + lane * 16);
    };
    auto tail = [&](auto self, auto pc) {
      constexpr int P = decltype(pc)::value;
      if constexpr (P == NP1) { if (wid == 0) { TS(6); TC(22); TS(22); } }
      auto hook = [&](int j) {
        if constexpr (P >= NP1) {
          const int i = (P - NP1) * XPP + (j - XOFF) / XGAP;
          if (j >= XOFF && (j - XOFF) % XGAP == 0 && (j - XOFF) / XGAP < XPP && i < 5) {
            xsend(i);
          }
        }
      };
      if constexpr (P == NP - XAT) xfetch();
      if constexpr (P == NP - 1) { failw = ctl[0]; xread(); }
      using SLc = std::integral_constant<int, (P + 1) & 3>;
      using MCc = std::integral_constant<int, (P < NP1) ? 4 : 2>;
      using MNc = std::integral_constant<int, (P + 1 < NP1) ? 4 : (P + 1 < NP) ? 2 : 0>;
      if constexpr (P & 1) period(f1, f0, SLc{}, MCc{}, MNc{}, hook); else period(f0, f1, SLc{}, MCc{}, MNc{}, hook);
      if constexpr (P < NP1) ENDP1(); else if constexpr (P + 2 < NP) ENDP();
      if constexpr (P + 1 < NP) self(self, std::integral_constant<int, P + 1>{});
    };
    tail(tail, std::integral_constant<int, NPL>{});
    bool fail = __builtin_amdgcn_readfirstlane(failw) != 0;
    {
      u64 t0 = 0;
      while (true) {
        unsigned bad = 0;
#pragma unroll
        for (int i = 0; i < 5; ++i) bad |= xp[i][3] ^ tagh;
        if (__builtin_expect(__all(bad < 0x10000u) || fail, 1)) break;
        const u64 now = __builtin_amdgcn_s_memrealtime();
        if (t0 == 0) t0 = now;
        if (now - t0 > DEADLINE) fail = true;
        xfetch();
        xread();
      }
    }
    asm volatile("" : "+v"(acc[1][3]), "+v"(acc[0][3]), "+v"(acc[1][2]), "+v"(acc[0][2]));
    if (wid == 0) { TS(1); TC(23); TS(23); }
    typedef unsigned u32x2 __attribute__((ext_vector_type(2)));
    auto xv = [&](int i) {
      const unsigned v = xp[i / 7][(i % 7) >> 1];
      return __uint_as_float(((i % 7) & 1) ? v & 0xffff0000u : v << 16);
    };
    char* obase = p.out + ((size_t)((bm2 * 2 + q) * 64 + wm * 32 + (lane >> 3)) * NN + bn * 128 + wn * 64) * 2 + (lane & 7) * 16;
#pragma unroll
    for (int m = 0; m < 2; ++m) {
#pragma unroll
      for (int n = 0; n < 4; ++n) {
        const f32x4 c0 = acc[m][n];
        const int i0 = m * 16 + n * 4;
        *(volatile LAS u32x2*)(size_t)(xl + (m * 16 + l15) * 144 + g * 8 + n * 32) = (u32x2){pk(c0[0] + xv(i0), c0[1] + xv(i0 + 1)), pk(c0[2] + xv(i0 + 2), c0[3] + xv(i0 + 3))};
      }
    }
    u32x4 ov[4];
#pragma unroll
    for (int i = 0; i < 4; ++i) ov[i] = *(volatile LAS u32x4*)(size_t)(xl + (i * 8 + (lane >> 3)) * 144 + (lane & 7) * 16);
#pragma unroll
    for (int i = 0; i < 4; ++i) {
      if (fail) ov[i] = (u32x4){0x7fc07fc0u, 0x7fc07fc0u, 0x7fc07fc0u, 0x7fc07fc0u};
      st16_sys(obase + (size_t)i * 8 * NN * 2, ov[i]);
    }
  } else if (wid == 4) {
    const int r = lane >> 4, c = lane & 15;
    unsigned voff[8];
#pragma unroll
    for (int u = 0; u < 8; ++u) voff[u] = (4 * u + r) * (NN * 2) + ((c ^ (r << 2) ^ (u & 3)) << 4) + 4096 - u * 1024;
    const char* base = p.w + bn * 256;
    auto issue = [&](const char* sb, unsigned la) { dma8k_v(sb, voff, la); dma8k_v(sb + 32 * NN * 2, voff, la + 8192); };
    auto src = [&](int code) { return base + (size_t)code * (64 * NN * 2); };
    if (early) BAR(); else __builtin_amdgcn_s_sleep(HOLD);
    issue(src(rank * 8 + 2 * (rot & 3) + q), lds0 + 4096);
    issue(src(rank * 8 + 2 * ((1 + rot) & 3) + q), lds0 + 4096 + SLOT);
    const int tab = sched_code(lane < NP ? lane : 0, rank, q, rot, bn & 3);
    WAIT_V(16);
    BAR();
    {
      const char* s4 = src(__builtin_amdgcn_readlane(tab, 4));
      const unsigned po = (lane >> 1) * (NN * 2) + (lane & 1) * 128;
      if (idx >= SKP0 && idx <= SKP1) WAIT_V(0);
      else { dma4(s4, po, lds0 + LDS_JNK); dma4(s4 + 32 * NN * 2, po, lds0 + LDS_JNK); WAIT_V(2); }
    }
    BAR();
    issue(src(rank * 8 + 2 * ((2 + rot) & 3) + q), lds0 + 4096 + 2 * SLOT);
    issue(src(rank * 8 + 2 * ((3 + rot) & 3) + q), lds0 + 4096 + 3 * SLOT);
    WAIT_V(16);
    BAR();
    issue(src(__builtin_amdgcn_readlane(tab, 4)), lds0 + 4096);
    const char* sb = src(__builtin_amdgcn_readlane(tab, 5));
    unsigned la = lds0 + 4096 + SLOT;
    for (int k = 2; k < NP - D; ++k) {
      dma8k_v(sb, voff, la);
      const char* sb2 = sb + 32 * NN * 2;
      unsigned la2 = la + 8192;
      sb = src(__builtin_amdgcn_readlane(tab, k + D + 1));
      la = lds0 + 4096 + ((k + D + 1) & 3) * SLOT;
      asm volatile("" : "+s"(sb), "+s"(la), "+s"(sb2), "+s"(la2));
      dma8k_v(sb2, voff, la2);
      WAIT_V(32);
      BAR();
    }
    WAIT_V(16); BAR();
    WAIT_V(0); BAR();
  } else {
    const int r = lane >> 3, c = lane & 7;
    unsigned voffx[8];
#pragma unroll
    for (int u = 0; u < 8; ++u) voffx[u] = (8 * u + r) * (KLOC * 2) + ((c ^ (((u & 1) << 2) | (r >> 1))) << 4) + 4096 - u * 1024;
    char* cnt = myws + WS_CNT + b * 128;
    volatile LAS unsigned* fl = (volatile LAS unsigned*)(size_t)(lds0 + LDS_FLG + lane * 4);
    if (wid == 5) {
      const u64 t_start = __builtin_amdgcn_s_memrealtime();
      const char* xb = p.x + (size_t)(rank * MLOC + bm2 * 128) * (KLOC * 2);
      auto issue_local = [&](int e) {
        const char* sb = xb + (2 * ((e + rot) & 3) + q) * 128;
        const unsigned la = lds0 + e * SLOT + W_TILE + 4096;
        dma8k_v(sb + q * (64 * KLOC * 2), voffx, la);
        dma8k_v(sb + (q ^ 1) * (64 * KLOC * 2), voffx, la + UNIT);
      };
      fl[0] = 0; ctl[0] = 0;
      if (early) BAR(); else __builtin_amdgcn_s_sleep(HOLD);
      dma4_sys(cnt, 0u, lds0 + LDS_CNT);
      issue_local(0); issue_local(1);
      const int tab = sched_code(lane < NP ? lane : 0, rank, q, rot, bn & 3);
      WAIT_V(16);
      const unsigned call = __builtin_amdgcn_readfirstlane(ctl[64]) + 1;
      const char* db = myws + (call & 1) * WS_BUF + (size_t)(bm2 * 2) * 8 * UNIT + 4096;
      const char* fp = myws + WS_FLAG + bm2 * 8192;
      const unsigned vlin = lane * 16;
      const bool nocheck = lane < 4 || lane >= NP;
      const unsigned fvo = tab * 128;
      const unsigned texp = PUSH0 + VIS + slot_time((tab & 7) * 4 + ((((((tab >> 3) - rank - 1) & 7) >> 1) - bm2) & 3));
      const unsigned aoffs = (tab >> 3) * (MLOC * KLOC * 2) + (tab & 7) * UNIT;
      int avail = 4;
      int since = 0;
      const unsigned fthr = nocheck ? 0u : call;
      const unsigned tlow = (unsigned)t_start;
      auto poll_fire = [&](int upto, unsigned now) {
        const u64 m = __ballot((lane >= avail) & (lane < NP) & ((lane <= upto) | (texp <= now - tlow)));
        asm volatile("s_mov_b32 m0, %2\n\ts_mov_b64 exec, %3\n\tglobal_load_lds_dword %0, %1 sc0 sc1\n\ts_mov_b64 exec, -1" ::"v"(fvo), "s"(fp), "s"(lds0 + LDS_FLG), "s"(m) : "memory");
        ++since;
      };
      auto poll_issue = [&](int upto) { poll_fire(upto, (unsigned)__builtin_amdgcn_s_memrealtime()); };
      auto poll_read = [&]() { avail = first_clear(*fl >= fthr); };
      WAIT_L0();
      BAR();
      st4_sys_l0(cnt, call);
      WAIT_V(1);
      BAR();
      issue_local(2);
      issue_local(3);
      poll_issue(FP0);
      WAIT_V(17);
      BAR();
      WAIT_V(0);
      poll_read();
      int na = 4;
      auto issue = [&]() {
        const char* sb = db + (unsigned)__builtin_amdgcn_readlane(aoffs, na);
        const unsigned la = lds0 + (na & 3) * SLOT + W_TILE + 4096;
        dma8k_lin(sb + q * (8 * UNIT), vlin, la);
        if (na < NP1) dma8k_lin(sb + (q ^ 1) * (8 * UNIT), vlin, la + UNIT);
        ++na; since = 0;
      };
      auto gate = [&](int k) {
        if (avail < k + DA + 1 + PTH && avail < NP) {
          poll_read();
          if (avail <= k + DA && avail > k + 1) {
            poll_issue(k + DA + RUP);
            WAIT_V(0);
            poll_read();
          }
          if (avail <= k + 1) {
            const u64 t0 = __builtin_amdgcn_s_memrealtime();
            while (true) {
              poll_issue(k + 2);
              WAIT_V(0);
              poll_read();
              if (avail > k + 1) break;
              if (__builtin_amdgcn_s_memrealtime() - t0 > DEADLINE) { avail = 64; *fl = ~0u; ctl[0] = 1; WAIT_L0(); break; }
              __builtin_amdgcn_s_sleep(1);
            }
          }
          if (avail < k + DA + 1 + PTH && avail < NP) poll_issue(avail);
        }
      };
      int kpoll = 0;
      const char* db0 = db + q * (8 * UNIT);
      const char* db1 = db + (q ^ 1) * (8 * UNIT);
      for (int k = 2; k < NP1 - DA; ++k) {
        if (na == k + DA && avail > na) {
          const unsigned o = __builtin_amdgcn_readlane(aoffs, na);
          const unsigned la = lds0 + (na & 3) * SLOT + W_TILE + 4096;
          const char* s0 = db0 + o; const char* s1 = db1 + o;
          unsigned la1 = la + UNIT;
          ++na;
          asm volatile("" : "+s"(s1), "+s"(la1), "+s"(na));
          if (k < kpoll) {
            dma8k_lin(s0, vlin, la);
            SB();
            const unsigned fv = *fl;
            SB();
            dma8k_lin(s1, vlin, la1);
            WAIT_V(16 * (DA - 1));
            BAR();
            avail = first_clear(fv >= fthr);
          } else if (avail < k + DA + 1 + PTH) {
            const unsigned now = (unsigned)__builtin_amdgcn_s_memrealtime();
            SB();
            dma8k_lin(s0, vlin, la);
            SB();
            poll_fire(avail, now); kpoll = k + 2;
            SB();
            dma8k_lin(s1, vlin, la1);
            WAIT_V(16 * (DA - 1) + 1);
            BAR();
          } else {
            dma8k_lin(s0, vlin, la);
            dma8k_lin(s1, vlin, la1);
            WAIT_V(16 * (DA - 1));
            BAR();
          }
          since = 0;
          continue;
        } else {
          gate(k);
          kpoll = k + 2;
          const int lim = k + DA + 1;
          const int top = avail < lim ? avail : lim;
          while (na < top) issue();
          const int ahead = na - 2 - k;
          if (ahead >= 2) WAIT_V(32); else if (ahead == 1) WAIT_V(16);
          else if (since >= 3) WAIT_V(3); else if (since == 2) WAIT_V(2); else if (since == 1) WAIT_V(1); else WAIT_V(0);
        }
        BAR();
      }
      for (int k = NP1 - DA; k < NP - 1; ++k) {
        const int lim = k + D + 1 > NP ? NP : k + D + 1;
        gate(k);
        const int top = avail < lim ? avail : lim;
        while (na < top) issue();
        const int ahead = na - 2 - k;
        if (ahead >= 2) WAIT_V(16); else if (ahead == 1) WAIT_V(8); else WAIT_V(0);
        BAR();
      }
    } else {
      const u64 t_start = __builtin_amdgcn_s_memrealtime();
      const int pj = idx >> 2;
      const int prb = ((((rank - (int)pd - 1) & 7) >> 1) - idx) & 3;
      if (pd == (unsigned)rank) return;
      const char* ps = p.x + (size_t)(pd * MLOC + prb * 128) * (KLOC * 2) + pj * 128;
      const unsigned stg = lds0 + LDS_X;
      auto stage = [&]() { dma8k_v(ps, voffx, stg + 4096); dma8k_v(ps + 64 * KLOC * 2, voffx, stg + UNIT + 4096); };
      u32x4 pv[16];
      unsigned call, cv;
      if (pj >= 2 && pj <= PJR) __builtin_amdgcn_s_sleep(5);
      if (pj <= PJR) {
        STAMP(st1);
        ld8k(ps, voffx, pv);
        asm volatile("global_load_dword %0, %1, %2 sc0 sc1" : "=v"(cv) : "v"(0u), "s"(cnt) : "memory");
        ld8k(ps + 64 * KLOC * 2, voffx, pv + 8);
        STAMP(st2);
        SB();
        char* pw = p.ws[0];
#pragma unroll
        for (int i = 1; i < NR; ++i) pw = pd == (unsigned)i ? p.ws[i] : pw;
        const char* pe0 = pw + (size_t)((rank * 8 + prb * 2) * 8 + pj) * UNIT + 4096;
        unsigned vl = lane * 16;
        asm volatile("" : "+s"(pe0));
        asm volatile("" : "+v"(vl));
        asm volatile("s_waitcnt vmcnt(0)" : "+v"(cv), "+v"(pv[0]), "+v"(pv[1]), "+v"(pv[2]), "+v"(pv[3]), "+v"(pv[4]), "+v"(pv[5]), "+v"(pv[6]), "+v"(pv[7]),
                     "+v"(pv[8]), "+v"(pv[9]), "+v"(pv[10]), "+v"(pv[11]), "+v"(pv[12]), "+v"(pv[13]), "+v"(pv[14]), "+v"(pv[15]) :: "memory");
        STAMP(st3);
        if (idx == 0) {
          const unsigned vo = vl + ((~cv & 1u) << 22);
          st8k_sys(pe0, vo, pv);
          st8k_sys(pe0 + 8 * UNIT, vo, pv + 8);
        }
        call = __builtin_amdgcn_readfirstlane(cv) + 1;
      } else {
        BAR();
        call = __builtin_amdgcn_readfirstlane(ctl[64]) + 1;
      }
      char* pw = p.ws[0];
#pragma unroll
      for (int i = 1; i < NR; ++i) pw = pd == (unsigned)i ? p.ws[i] : pw;
      const u64 t_push = t_start + PUSH0 + slot_time(idx);
      const char* pd0 = pw + (call & 1) * WS_BUF + (size_t)((rank * 8 + prb * 2) * 8 + pj) * UNIT + 4096;
      char* pflag = pw + WS_FLAG + ((prb * 8 + rank) * 8 + pj) * 128;
      bool starving = true;
      if (pj != 0) {
        const int tab = sched_code(lane < NP ? lane : 0, rank, q, rot, bn & 3);
        int k = pj <= PJR ? -1 : 0;
        auto run_until = [&](u64 t) {
          for (;; ++k) {
            starving = true;
            const int code = __builtin_amdgcn_readlane(tab, k + 1);
            if (k >= NP1 - 2 || (k >= 3 && (code & 7) >= pj)) break;
            bool go = false;
            while (true) {
              starving = k >= 3 && (unsigned)__builtin_amdgcn_readlane(*fl, k + 1) < call;
              if (__builtin_amdgcn_s_memrealtime() + (starving ? 0 : k < 5 ? PE1 : PEAR) >= t) { go = true; break; }
              if (!starving) break;
              __builtin_amdgcn_s_sleep(2);
            }
            if (go) break;
            BAR();
          }
        };
        if (pj > PJR) {
          const int kst = pj < PJL ? KST : pj < PJM ? 5 - q : 8;
          for (; k < kst; ++k) BAR();
          __builtin_amdgcn_s_sleep(KSL);
          stage();
          BAR(); ++k;
          if (kst > 3) { BAR(); ++k; }
          WAIT_V(0);
#pragma unroll
          for (int i = 0; i < 16; ++i) pv[i] = *(volatile LAS u32x4*)(size_t)(stg + i * 1024 + lane * 16);
        }
        run_until(t_push);
        while (k < 5 && !starving && __builtin_amdgcn_s_memrealtime() < t_push) __builtin_amdgcn_s_sleep(1);
        if (k <= KNB) starving = true;
      } else {
        while (idx != 0 && __builtin_amdgcn_s_memrealtime() < t_push) __builtin_amdgcn_s_sleep(1);
      }
      STAMP(st4);
      if (idx != 0) {
        st8k_sys(pd0, lane * 16, pv);
        st8k_sys(pd0 + 8 * UNIT, lane * 16, pv + 8);
      }
      TS(5);
      if (!starving || early) BAR();
      WAIT_V(0);
      asm volatile("s_mov_b64 exec, 0xffffffff\n\tglobal_store_dword %0, %1, off sc0 sc1\n\ts_mov_b64 exec, -1" ::"v"(pflag + lane * 4), "v"(call) : "memory");
      TS(7);
      BAR(); BAR();
    }
  }
}
}

extern "C" void kernel_launch(int rank, int nranks, ncclComm_t comm, hipStream_t stream,
                              void* const* d_in, const int* in_sizes, int n_in,
                              void* d_out, int out_size, const int* in_dtypes, int out_dtype,
                              void* const* d_out_all, void* const* d_ws_all, size_t ws_bytes) {
  if (nranks != NR || ws_bytes < WS_END || (unsigned)rank >= (unsigned)NR) { fprintf(stderr, "bad config\n"); return; }
  static bool init = false;
  if (!init) {
    HIP_CHECK(hipFuncSetAttribute((const void*)a2a_gemm, hipFuncAttributeMaxDynamicSharedMemorySize, LDS_BYTES));
    init = true;
  }
  Params p;
  p.x = (const char*)d_in[0];
  p.w = (const char*)d_in[1];
  p.out = (char*)d_out;
  for (int i = 0; i < NR; ++i) p.ws[i] = (char*)d_ws_all[i];
  p.myws = (char*)d_ws_all[rank];
  p.rank = rank;
  p.pad = 0;
  hipLaunchKernelGGL(a2a_gemm, dim3(256), dim3(448), LDS_BYTES, stream, p);
  HIP_CHECK(hipGetLastError());
}
```

```cpp
#include <hip/hip_runtime.h>
#include <rccl/rccl.h>
#include <cstdio>
#include <type_traits>

#define HIP_CHECK(expr) do { hipError_t _e = (expr); if (_e != hipSuccess) { \
    fprintf(stderr, "HIP error %s at %s:%d\n", hipGetErrorString(_e), __FILE__, __LINE__); \
    return; } } while (0)

typedef unsigned u32x2 __attribute__((ext_vector_type(2)));
typedef unsigned u32x4 __attribute__((ext_vector_type(4)));
typedef float f32x4 __attribute__((ext_vector_type(4)));
typedef __bf16 bf16x2 __attribute__((ext_vector_type(2)));
typedef __bf16 bf16x4 __attribute__((ext_vector_type(4)));
typedef __bf16 bf16x8 __attribute__((ext_vector_type(8)));
typedef unsigned long long u64;
#define LAS __attribute__((address_space(3)))

namespace {
constexpr int NR = 8, MLOC = 512, KLOC = 512, NN = 4096;
constexpr int W_TILE = 128 * 64 * 2;
constexpr int UNIT = 64 * 64 * 2;
constexpr int SLOT = W_TILE + 2 * UNIT;
constexpr int F = 8;
constexpr int NP1 = (64 - F) / 2;
constexpr int NP = NP1 + F;
constexpr int DW = 3, DA = 2;
constexpr int XB = 5 * 1024;
constexpr int LDS_X = 4 * SLOT;
constexpr int LDS_CTL = LDS_X + 4 * XB;
constexpr int LDS_CNT = LDS_CTL + 256;
constexpr int LDS_FLG = LDS_CNT + 256;
constexpr int LDS_JNK = LDS_FLG + 256;
constexpr int LDS_BYTES = LDS_JNK + 256;
constexpr size_t WS_BUF = (size_t)NR * MLOC * KLOC * 2;
constexpr size_t WS_XCH = 2 * WS_BUF;
constexpr size_t WS_FLAG = WS_XCH + 256 * 4 * XB;
constexpr size_t WS_CNT = WS_FLAG + 256 * 128;
constexpr size_t WS_END = WS_CNT + 256 * 128;
static_assert(WS_BUF == (1u << 22), "");
constexpr u64 DEADLINE = 50000000ull;
constexpr int PUSH0 = 58;
constexpr int NS1 = 12, TAU1 = 126, TAU = 148;
constexpr int VIS = 200;
constexpr int EARLYN = 3;
constexpr int FREEN = 12;
constexpr int PTH = 3;
__device__ __forceinline__ int slot_time(int s) { return s < NS1 ? (s * TAU1 >> 2) : ((NS1 * TAU1 + (s - NS1) * TAU) >> 2); }

struct Params {
  const char* x;
  const char* w;
  char* out;
  char* myws;
  int rank;
  int pad;
  char* ws[NR];
};

#define WAIT_V(n) asm volatile("s_waitcnt vmcnt(%0)" ::"n"(n) : "memory")
#define WAIT_L0() __builtin_amdgcn_s_waitcnt(0xc07f)
#define BAR() __builtin_amdgcn_s_barrier()
#define SB() __builtin_amdgcn_sched_barrier(0)
#define NOW() __builtin_amdgcn_s_memrealtime()
#define MFMA(c, x, y) asm volatile("v_mfma_f32_16x16x32_bf16 %0, %1, %2, %0" : "+v"(c) : "v"(x), "v"(y))

__device__ __forceinline__ void st16_sys(const char* sbase, unsigned voff, u32x4 v) {
  asm volatile("global_store_dwordx4 %0, %1, %2 sc0 sc1\n\ts_nop 1" ::"v"(voff), "v"(v), "s"(sbase) : "memory");
}
__device__ __forceinline__ void st_line_sys(const char* sbase, unsigned lane4, unsigned v) {
  asm volatile("s_mov_b32 exec_hi, 0\n\tglobal_store_dword %0, %1, %2 sc0 sc1\n\ts_mov_b32 exec_hi, -1" ::"v"(lane4), "v"(v), "s"(sbase) : "memory");
}
#define GL_(o) "global_load_lds_dwordx4 %0, %1 offset:" #o " sc0 sc1\n\t"
__device__ __forceinline__ void dma8k_lin(const char* sbase, unsigned voff, unsigned ldsaddr) {
  asm volatile("s_mov_b32 m0, %2\n\ts_nop 0\n\t" GL_(-4096) GL_(-3072) GL_(-2048) GL_(-1024) GL_(0) GL_(1024) GL_(2048) GL_(3072)
               ::"v"(voff), "s"(sbase), "s"(ldsaddr) : "memory");
}
__device__ __forceinline__ void dma5k_lin(const char* sbase, unsigned voff, unsigned ldsaddr) {
  asm volatile("s_mov_b32 m0, %2\n\ts_nop 0\n\t" GL_(-4096) GL_(-3072) GL_(-2048) GL_(-1024) GL_(0) ::"v"(voff), "s"(sbase), "s"(ldsaddr) : "memory");
}
#define GV_(i, o) "global_load_lds_dwordx4 %" #i ", %8 offset:" #o "\n\t"
__device__ __forceinline__ void dma8k_v(const char* sbase, const unsigned (&v)[8], unsigned ldsaddr) {
  asm volatile("s_mov_b32 m0, %9\n\ts_nop 0\n\t" GV_(0, -4096) GV_(1, -3072) GV_(2, -2048) GV_(3, -1024) GV_(4, 0) GV_(5, 1024) GV_(6, 2048) GV_(7, 3072)
               ::"v"(v[0]), "v"(v[1]), "v"(v[2]), "v"(v[3]), "v"(v[4]), "v"(v[5]), "v"(v[6]), "v"(v[7]), "s"(sbase), "s"(ldsaddr) : "memory");
}
#define GS_(i, o) "global_store_dwordx4 %8, %" #i ", %9 offset:" #o " sc0 sc1\n\t"
__device__ __forceinline__ void st8k_sys(const char* sbase, unsigned voff, const u32x4* v) {
  asm volatile(GS_(0, -4096) GS_(1, -3072) GS_(2, -2048) GS_(3, -1024) GS_(4, 0) GS_(5, 1024) GS_(6, 2048) GS_(7, 3072) "s_nop 1"
               ::"v"(v[0]), "v"(v[1]), "v"(v[2]), "v"(v[3]), "v"(v[4]), "v"(v[5]), "v"(v[6]), "v"(v[7]), "v"(voff), "s"(sbase) : "memory");
}
#define GR_(i, a, o) "global_load_dwordx4 %" #i ", %" #a ", %16 offset:" #o "\n\t"
__device__ __forceinline__ void ld8k(const char* sbase, const unsigned (&v)[8], u32x4* d) {
  asm volatile(GR_(0, 8, -4096) GR_(1, 9, -3072) GR_(2, 10, -2048) GR_(3, 11, -1024) GR_(4, 12, 0) GR_(5, 13, 1024) GR_(6, 14, 2048) GR_(7, 15, 3072)
               : "=&v"(d[0]), "=&v"(d[1]), "=&v"(d[2]), "=&v"(d[3]), "=&v"(d[4]), "=&v"(d[5]), "=&v"(d[6]), "=&v"(d[7])
               : "v"(v[0]), "v"(v[1]), "v"(v[2]), "v"(v[3]), "v"(v[4]), "v"(v[5]), "v"(v[6]), "v"(v[7]), "s"(sbase) : "memory");
}
__device__ __forceinline__ void dma4_sys(const char* sbase, unsigned voff, unsigned ldsaddr, u64 mask) {
  asm volatile("s_mov_b32 m0, %2\n\ts_mov_b64 exec, %3\n\tglobal_load_lds_dword %0, %1 sc0 sc1\n\ts_mov_b64 exec, -1"
               ::"v"(voff), "s"(sbase), "s"(ldsaddr), "s"(mask) : "memory");
}
__device__ __forceinline__ void touch_w(const char* sbase, int lane, unsigned ldsjunk) {
  const unsigned po = (lane >> 1) * (NN * 2) + (lane & 1) * 128;
  asm volatile("s_mov_b32 m0, %3\n\ts_nop 0\n\tglobal_load_lds_dword %0, %1\n\tglobal_load_lds_dword %0, %2"
               ::"v"(po), "s"(sbase), "s"(sbase + 32 * NN * 2), "s"(ldsjunk) : "memory");
}
__device__ __forceinline__ int first_clear(bool c) {
  const u64 m = ~__ballot(c);
  return __builtin_amdgcn_readfirstlane(m ? __builtin_ctzll(m) : 64);
}
__device__ __forceinline__ unsigned pk(float a, float b) { return __builtin_bit_cast(unsigned, (bf16x2){(__bf16)a, (__bf16)b}); }

__device__ __forceinline__ int sched_code(int e, int rank, int q, int rot) {
  if (e < 4) return rank * 8 + 2 * ((e + rot) & 3) + q;
  int u;
  if (e < NP1) {
    const int u0 = 2 * (e - 4) + q;
    const int r = u0 / 7;
    const int first = 7 * r + ((7 * r + q) & 1);
    int last = 7 * r + 6; if (last > 55 - F) last = 55 - F;
    const int c = (last - first) / 2 + 1;
    u = first + 2 * (((u0 - first) / 2 + rot) % c);
  } else {
    u = 56 - F + ((e - NP1 + rot * 2 + q) % F);
  }
  const int r = u / 7, i = u - 7 * r;
  return ((rank + 1 + i) & 7) * 8 + r;
}

struct Fr { bf16x8 a[2][4]; bf16x4 b[2][4][2]; };

__global__ __launch_bounds__(448, 2) void a2a_gemm(Params p) {
  extern __shared__ __attribute__((aligned(1024))) char smem[];
  const unsigned lds0 = (unsigned)(size_t)(LAS char*)smem;
  const int b = blockIdx.x;
  const int wid = __builtin_amdgcn_readfirstlane(threadIdx.x >> 6);
  const int lane = threadIdx.x & 63;
  const int xcd = b & 7, idx = b >> 3;
  const int q = xcd >> 2, bn = (xcd & 3) * 8 + (idx & 7), bm2 = idx >> 3;
  const unsigned pd = (unsigned)(b + idx) & 7u;
  const char *px = p.x, *pwm = p.w, *pout = p.out, *myws = p.myws, *peer = p.ws[pd];
  int rank = p.rank;
  asm volatile("" : "+s"(px), "+s"(pwm), "+s"(pout), "+s"(myws), "+s"(peer), "+s"(rank));
  const bool early = idx < EARLYN && pd != (unsigned)rank;
  volatile LAS unsigned* ctl = (volatile LAS unsigned*)(size_t)(lds0 + LDS_CTL);

  if (wid < 4) {
    const int wm = wid >> 1, wn = wid & 1;
    const int l15 = lane & 15, g = lane >> 4, qq = l15 >> 2, pp = lane & 3;
    const unsigned aoff = W_TILE + (wm * 32 + l15) * 128 + ((g ^ (l15 >> 1)) << 4);
    const unsigned boff = (8 * g + qq) * 256 + (((wn * 8 + (pp >> 1)) ^ (qq << 2) ^ ((2 * g) & 3)) << 4) + 8 * (pp & 1);
    f32x4 acc[4][4];
#pragma unroll
    for (int m = 0; m < 4; ++m)
#pragma unroll
      for (int n = 0; n < 4; ++n) { acc[m][n] = (f32x4){0.f, 0.f, 0.f, 0.f}; asm volatile("" : "+v"(acc[m][n])); }
    unsigned ra[2][2], rb[2][8];
#pragma unroll
    for (int hh = 0; hh < 2; ++hh) {
#pragma unroll
      for (int k2 = 0; k2 < 2; ++k2) { ra[hh][k2] = lds0 + hh * 65536 + (aoff ^ (k2 << 6)); asm volatile("" : "+v"(ra[hh][k2])); }
#pragma unroll
      for (int u = 0; u < 8; ++u) { rb[hh][u] = lds0 + hh * 65536 + (boff ^ (u << 4)); asm volatile("" : "+v"(rb[hh][u])); }
    }
    auto read = [&](Fr& f, int so, int mt, int rr) {
      const int hh = so / 65536, o = so % 65536;
      if (rr >= 4 && rr < 4 + mt * 2) {
        const int mm = (rr - 4) >> 1, kk = rr & 1;
        f.a[kk][mm] = *(const LAS bf16x8*)(size_t)(ra[hh][kk] + o + (mm >> 1) * UNIT + (mm & 1) * 2048);
      } else {
        const int x = rr < 4 ? rr : rr - mt * 2, nn = x >> 2, kk = (x >> 1) & 1, t = x & 1;
        f.b[kk][nn][t] = __builtin_amdgcn_ds_read_tr16_b64_v4bf16((LAS bf16x4*)(size_t)(rb[hh][nn * 2 + t] + o + t * 1024 + kk * 8192));
      }
    };
    auto period = [&](Fr& cur, Fr& nxt, auto pc, auto hook) {
      constexpr int P = decltype(pc)::value;
      constexpr int MC = P < NP1 ? 4 : 2, MN = P + 1 < NP1 ? 4 : 2;
      constexpr int NM = MC * 8, NRD = P + 1 < NP ? 16 + MN * 2 : 0, PER = MC == 4 ? 1 : 2;
      constexpr int IB = P + 2 >= NP ? -1 : MC == 4 ? NM - 3 : NM - 1;
      SB();
#pragma unroll
      for (int i = 0; i < NM; ++i) {
        const int n = i / (MC * 2), m = (i >> 1) % MC, k2 = i & 1;
        bf16x8 wv = __builtin_shufflevector(cur.b[k2][n][0], cur.b[k2][n][1], 0, 1, 2, 3, 4, 5, 6, 7);
        MFMA(acc[m][n], wv, cur.a[k2][m]);
#pragma unroll
        for (int rr = i * PER; rr < i * PER + PER; ++rr)
          if (rr < NRD) read(nxt, ((P + 1) & 3) * SLOT, MN, rr);
        hook(i);
        if (i == IB) { SB(); WAIT_L0(); BAR(); }
        SB();
      }
    };
    auto nohook = [](int) {};
    Fr f0, f1;
    if (early) BAR();
    BAR();
#pragma unroll
    for (int rr = 0; rr < 24; ++rr) read(f0, 0, 4, rr);
    const unsigned call = __builtin_amdgcn_readfirstlane(ctl[64]) + 1;
    WAIT_L0(); BAR();
    constexpr int NPL = (NP1 - 1) / 4 * 4;
    for (int pr = 0; pr < NPL; pr += 4) {
      period(f0, f1, std::integral_constant<int, 0>{}, nohook);
      period(f1, f0, std::integral_constant<int, 1>{}, nohook);
      period(f0, f1, std::integral_constant<int, 2>{}, nohook);
      period(f1, f0, std::integral_constant<int, 3>{}, nohook);
    }
    const unsigned tagh = call << 16;
    const char* xsrc = myws + WS_XCH + (size_t)(b * 4 + wid) * XB + 4096;
    const char* xdst = myws + WS_XCH + (size_t)((b ^ 4) * 4 + wid) * XB;
    const unsigned xl = lds0 + LDS_X + wid * XB;
    u32x4 xp[5];
    unsigned failw = 0;
    auto xsend = [&](int i) {
      auto val = [&](int v) { return v < 32 ? acc[2 + (v >> 4)][(v >> 2) & 3][v & 3] : 0.f; };
      const u32x4 d = {pk(val(7 * i), val(7 * i + 1)), pk(val(7 * i + 2), val(7 * i + 3)), pk(val(7 * i + 4), val(7 * i + 5)),
                       (pk(val(7 * i + 6), 0.f) & 0xffffu) | tagh};
      st16_sys(xdst + i * 1024, lane * 16, d);
    };
    auto xfetch = [&]() { dma5k_lin(xsrc, lane * 16, xl + 4096); };
    auto xread = [&]() {
      WAIT_V(0);
#pragma unroll
      for (int i = 0; i < 5; ++i) xp[i] = *(volatile LAS u32x4*)(size_t)(xl + i * 1024 + lane * 16);
    };
    auto tail = [&](auto self, auto pc) {
      constexpr int P = decltype(pc)::value;
      auto hook = [&](int j) { if (P == NP1 && j >= 10 && j < 15) xsend(j - 10); };
      if constexpr (P == NP - 3) xfetch();
      if constexpr (P == NP - 1) { failw = ctl[0]; xread(); }
      if constexpr (P & 1) period(f1, f0, pc, hook); else period(f0, f1, pc, hook);
      if constexpr (P + 1 < NP) self(self, std::integral_constant<int, P + 1>{});
    };
    tail(tail, std::integral_constant<int, NPL>{});
    bool fail = __builtin_amdgcn_readfirstlane(failw) != 0;
    {
      u64 t0 = 0;
      while (true) {
        unsigned bad = 0;
#pragma unroll
        for (int i = 0; i < 5; ++i) bad |= xp[i][3] ^ tagh;
        if (__builtin_expect(__all(bad < 0x10000u) || fail, 1)) break;
        const u64 now = NOW();
        if (t0 == 0) t0 = now;
        if (now - t0 > DEADLINE) fail = true;
        xfetch();
        xread();
      }
    }
    asm volatile("" : "+v"(acc[1][3]), "+v"(acc[0][3]), "+v"(acc[1][2]), "+v"(acc[0][2]));
    auto xv = [&](int i) {
      const unsigned v = xp[i / 7][(i % 7) >> 1];
      return __uint_as_float(((i % 7) & 1) ? v & 0xffff0000u : v << 16);
    };
#pragma unroll
    for (int m = 0; m < 2; ++m)
#pragma unroll
      for (int n = 0; n < 4; ++n) {
        const f32x4 c0 = acc[m][n];
        const int i0 = m * 16 + n * 4;
        *(volatile LAS u32x2*)(size_t)(xl + (m * 16 + l15) * 144 + g * 8 + n * 32) =
            (u32x2){pk(c0[0] + xv(i0), c0[1] + xv(i0 + 1)), pk(c0[2] + xv(i0 + 2), c0[3] + xv(i0 + 3))};
      }
    u32x4 ov[4];
#pragma unroll
    for (int i = 0; i < 4; ++i) ov[i] = *(volatile LAS u32x4*)(size_t)(xl + (i * 8 + (lane >> 3)) * 144 + (lane & 7) * 16);
    const char* obase = pout + ((size_t)((bm2 * 2 + q) * 64 + wm * 32) * NN + bn * 128 + wn * 64) * 2;
#pragma unroll
    for (int i = 0; i < 4; ++i) {
      if (fail) ov[i] = (u32x4){0x7fc07fc0u, 0x7fc07fc0u, 0x7fc07fc0u, 0x7fc07fc0u};
      st16_sys(obase + i * 8 * NN * 2, (lane >> 3) * (NN * 2) + (lane & 7) * 16, ov[i]);
    }
  } else if (wid == 4) {
    const int r = lane >> 4, c = lane & 15;
    unsigned voff[8];
#pragma unroll
    for (int u = 0; u < 8; ++u) voff[u] = (4 * u + r) * (NN * 2) + ((c ^ (r << 2) ^ (u & 3)) << 4) + 4096 - u * 1024;
    const char* base = pwm + bn * 256;
    auto issue = [&](const char* sb, unsigned la) { dma8k_v(sb, voff, la); dma8k_v(sb + 32 * NN * 2, voff, la + 8192); };
    auto src = [&](int code) { return base + (size_t)code * (64 * NN * 2); };
    auto issue_local = [&](int e) { issue(src(rank * 8 + 2 * ((e + bm2) & 3) + q), lds0 + 4096 + e * SLOT); };
    if (early) BAR(); else __builtin_amdgcn_s_sleep(12);
    issue_local(0); issue_local(1);
    const int tab = sched_code(lane < NP ? lane : 0, rank, q, bm2);
    WAIT_V(16);
    BAR();
    touch_w(src(__builtin_amdgcn_readlane(tab, 4)), lane, lds0 + LDS_JNK);
    WAIT_V(2);
    BAR();
    issue_local(2); issue_local(3);
    WAIT_V(16);
    BAR();
    issue(src(__builtin_amdgcn_readlane(tab, 4)), lds0 + 4096);
    const char* sb = src(__builtin_amdgcn_readlane(tab, 5));
    unsigned la = lds0 + 4096 + SLOT;
    for (int k = 2; k < NP - DW; ++k) {
      dma8k_v(sb, voff, la);
      const char* sb2 = sb + 32 * NN * 2;
      unsigned la2 = la + 8192;
      sb = src(__builtin_amdgcn_readlane(tab, k + DW + 1));
      la = lds0 + 4096 + ((k + DW + 1) & 3) * SLOT;
      asm volatile("" : "+s"(sb), "+s"(la), "+s"(sb2), "+s"(la2));
      dma8k_v(sb2, voff, la2);
      WAIT_V(32);
      BAR();
    }
    WAIT_V(16); BAR();
    WAIT_V(0); BAR();
  } else {
    const u64 t_start = NOW();
    const int r = lane >> 3, c = lane & 7;
    unsigned voffx[8];
#pragma unroll
    for (int u = 0; u < 8; ++u) voffx[u] = (8 * u + r) * (KLOC * 2) + ((c ^ (((u & 1) << 2) | (r >> 1))) << 4) + 4096 - u * 1024;
    const char* cnt = myws + WS_CNT + b * 128;
    volatile LAS unsigned* fl = (volatile LAS unsigned*)(size_t)(lds0 + LDS_FLG + lane * 4);
    if (wid == 5) {
      const char* xb = px + (size_t)(rank * MLOC + bm2 * 128) * (KLOC * 2);
      auto issue_local = [&](int e) {
        const char* sb = xb + (2 * ((e + bm2) & 3) + q) * 128;
        const unsigned la = lds0 + e * SLOT + W_TILE + 4096;
        dma8k_v(sb + q * (64 * KLOC * 2), voffx, la);
        dma8k_v(sb + (q ^ 1) * (64 * KLOC * 2), voffx, la + UNIT);
      };
      *fl = 0; ctl[0] = 0;
      if (early) BAR(); else __builtin_amdgcn_s_sleep(12);
      dma4_sys(cnt, 0u, lds0 + LDS_CNT, 1);
      issue_local(0); issue_local(1);
      const int tab = sched_code(lane < NP ? lane : 0, rank, q, bm2);
      WAIT_V(16);
      const unsigned call = __builtin_amdgcn_readfirstlane(ctl[64]) + 1;
      const char* db0 = myws + (call & 1) * WS_BUF + (size_t)(bm2 * 2 + q) * 8 * UNIT + 4096;
      const char* db1 = myws + (call & 1) * WS_BUF + (size_t)(bm2 * 2 + (q ^ 1)) * 8 * UNIT + 4096;
      const char* fp = myws + WS_FLAG + bm2 * 8192;
      const unsigned vlin = lane * 16;
      const unsigned fvo = tab * 128;
      const unsigned texp = PUSH0 + VIS + slot_time((tab & 7) * 4 + ((((((tab >> 3) - rank - 1) & 7) >> 1) - bm2) & 3));
      const unsigned aoffs = (tab >> 3) * (MLOC * KLOC * 2) + (tab & 7) * UNIT;
      const unsigned fthr = lane < 4 || lane >= NP ? 0u : call;
      int avail = 4;
      int since = 0;
      auto poll_fire = [&](int upto, u64 now) {
        dma4_sys(fp, fvo, lds0 + LDS_FLG, __ballot((lane >= avail) & (lane < NP) & ((lane <= upto) | (texp <= (unsigned)(now - t_start)))));
        ++since;
      };
      auto poll_read = [&]() { avail = first_clear(*fl >= fthr); };
      WAIT_L0();
      BAR();
      st_line_sys(cnt, lane * 4, call);
      WAIT_V(1);
      BAR();
      issue_local(2); issue_local(3);
      poll_fire(5, NOW());
      WAIT_V(17);
      BAR();
      WAIT_V(0);
      poll_read();
      int na = 4;
      auto issue = [&]() {
        const unsigned o = __builtin_amdgcn_readlane(aoffs, na);
        const unsigned la = lds0 + (na & 3) * SLOT + W_TILE + 4096;
        dma8k_lin(db0 + o, vlin, la);
        if (na < NP1) dma8k_lin(db1 + o, vlin, la + UNIT);
        ++na; since = 0;
      };
      bool pf = true;
      auto gate = [&](int k) {
        if (avail < k + DA + 1 + PTH && avail < NP) {
          poll_read();
          if (avail <= k + DA && avail > k + 1) {
            poll_fire(k + DA + 1, NOW());
            WAIT_V(0);
            poll_read();
          }
          if (avail <= k + 1) {
            const u64 t0 = NOW();
            while (true) {
              poll_fire(k + 2, NOW());
              WAIT_V(0);
              poll_read();
              if (avail > k + 1) break;
              if (k == 3 && pf) {
                touch_w(pwm + bn * 256 + (size_t)__builtin_amdgcn_readlane(tab, 8 - q) * (64 * NN * 2), lane, lds0 + LDS_JNK);
                pf = false;
              }
              if (NOW() - t0 > DEADLINE) { avail = 64; *fl = ~0u; ctl[0] = 1; WAIT_L0(); break; }
              __builtin_amdgcn_s_sleep(1);
            }
          }
          return avail < k + DA + 1 + PTH && avail < NP;
        }
        return false;
      };
      int kpoll = 0;
      for (int k = 2; k < NP1 - DA; ++k) {
        if (na == k + DA && avail > na) {
          const unsigned o = __builtin_amdgcn_readlane(aoffs, na);
          const unsigned la = lds0 + (na & 3) * SLOT + W_TILE + 4096;
          const char* s0 = db0 + o; const char* s1 = db1 + o;
          unsigned la1 = la + UNIT;
          ++na;
          asm volatile("" : "+s"(s1), "+s"(la1), "+s"(na));
          if (k < kpoll) {
            dma8k_lin(s0, vlin, la);
            SB();
            const unsigned fv = *fl;
            SB();
            dma8k_lin(s1, vlin, la1);
            WAIT_V(16);
            BAR();
            avail = first_clear(fv >= fthr);
          } else if (avail < k + DA + 1 + PTH) {
            const u64 now = NOW();
            SB();
            dma8k_lin(s0, vlin, la);
            SB();
            poll_fire(avail, now); kpoll = k + 2;
            SB();
            dma8k_lin(s1, vlin, la1);
            WAIT_V(17);
            BAR();
          } else {
            dma8k_lin(s0, vlin, la);
            dma8k_lin(s1, vlin, la1);
            WAIT_V(16);
            BAR();
          }
          since = 0;
          continue;
        }
        const bool want = gate(k);
        kpoll = k + 2;
        const int top = avail < k + DA + 1 ? avail : k + DA + 1;
        while (na < top) issue();
        if (want) { if (na == k + 2) __builtin_amdgcn_s_sleep(4); poll_fire(avail + 1, NOW()); }
        if (na > k + 2) { if (since) WAIT_V(17); else WAIT_V(16); }
        else if (since >= 3) WAIT_V(3); else if (since == 2) WAIT_V(2); else if (since == 1) WAIT_V(1); else WAIT_V(0);
        BAR();
      }
      for (int k = NP1 - DA; k < NP - 1; ++k) {
        if (gate(k)) poll_fire(avail, NOW());
        const int lim = k + DW + 1 > NP ? NP : k + DW + 1;
        const int top = avail < lim ? avail : lim;
        while (na < top) issue();
        const int ahead = na - 2 - k;
        if (ahead >= 2) WAIT_V(16); else if (ahead == 1) WAIT_V(8); else WAIT_V(0);
        BAR();
      }
    } else {
      if (pd == (unsigned)rank) return;
      const int pj = idx >> 2;
      const int prb = ((((rank - (int)pd - 1) & 7) >> 1) - idx) & 3;
      const char* ps = px + (size_t)(pd * MLOC + prb * 128) * (KLOC * 2) + pj * 128;
      const unsigned dof = ((rank * 8 + prb * 2) * 8 + pj) * UNIT + 4096;
      const u64 t_push = t_start + PUSH0 + slot_time(idx);
      u32x4 pv[16];
      unsigned call;
      bool hold = true;
      if (idx < FREEN) {
        unsigned cv;
        if (pj >= 2) __builtin_amdgcn_s_sleep(5);
        ld8k(ps, voffx, pv);
        asm volatile("global_load_dword %0, %1, %2 sc0 sc1" : "=v"(cv) : "v"(0u), "s"(cnt) : "memory");
        ld8k(ps + 64 * KLOC * 2, voffx, pv + 8);
        SB();
        const char* pe = peer + dof;
        unsigned vl = lane * 16;
        asm volatile("" : "+s"(pe));
        asm volatile("" : "+v"(vl));
        asm volatile("s_waitcnt vmcnt(0)" : "+v"(cv), "+v"(pv[0]), "+v"(pv[1]), "+v"(pv[2]), "+v"(pv[3]), "+v"(pv[4]), "+v"(pv[5]), "+v"(pv[6]), "+v"(pv[7]),
                     "+v"(pv[8]), "+v"(pv[9]), "+v"(pv[10]), "+v"(pv[11]), "+v"(pv[12]), "+v"(pv[13]), "+v"(pv[14]), "+v"(pv[15]) :: "memory");
        while (idx != 0 && NOW() < t_push) __builtin_amdgcn_s_sleep(1);
        const unsigned vo = vl + ((~cv & 1u) << 22);
        st8k_sys(pe, vo, pv);
        st8k_sys(pe + 8 * UNIT, vo, pv + 8);
        call = __builtin_amdgcn_readfirstlane(cv) + 1;
        hold = !early;
      } else {
        const int tab = sched_code(lane < NP ? lane : 0, rank, q, bm2);
        const unsigned stg = lds0 + LDS_X;
        BAR();
        call = __builtin_amdgcn_readfirstlane(ctl[64]) + 1;
        dma8k_v(ps, voffx, stg + 4096); dma8k_v(ps + 64 * KLOC * 2, voffx, stg + UNIT + 4096);
        BAR();
        WAIT_V(0);
#pragma unroll
        for (int i = 0; i < 16; ++i) pv[i] = *(volatile LAS u32x4*)(size_t)(stg + i * 1024 + lane * 16);
        const char* pe = peer + (call & 1) * WS_BUF + dof;
        int k = 1;
        for (;; ++k) {
          hold = true;
          if (k >= NP1 - 2 || (k >= 3 && (__builtin_amdgcn_readlane(tab, k + 1) & 7) >= pj)) break;
          bool go;
          while (true) {
            hold = k >= 3 && (unsigned)__builtin_amdgcn_readlane(*fl, k + 1) < call;
            go = NOW() + (hold ? 0 : k < 5 ? 50 : 20) >= t_push;
            if (go || !hold) break;
            __builtin_amdgcn_s_sleep(2);
          }
          if (go) break;
          BAR();
        }
        if (k < 5) { while (!hold && NOW() < t_push) __builtin_amdgcn_s_sleep(1); hold = true; }
        st8k_sys(pe, lane * 16, pv);
        st8k_sys(pe + 8 * UNIT, lane * 16, pv + 8);
      }
      if (!hold) BAR();
      WAIT_V(0);
      st_line_sys(peer + WS_FLAG + ((prb * 8 + rank) * 8 + pj) * 128, lane * 4, call);
      BAR(); BAR();
    }
  }
}
}

extern "C" void kernel_launch(int rank, int nranks, ncclComm_t comm, hipStream_t stream,
                              void* const* d_in, const int* in_sizes, int n_in,
                              void* d_out, int out_size, const int* in_dtypes, int out_dtype,
                              void* const* d_out_all, void* const* d_ws_all, size_t ws_bytes) {
  if (nranks != NR || ws_bytes < WS_END || (unsigned)rank >= (unsigned)NR) { fprintf(stderr, "bad config\n"); return; }
  static bool init = false;
  if (!init) {
    HIP_CHECK(hipFuncSetAttribute((const void*)a2a_gemm, hipFuncAttributeMaxDynamicSharedMemorySize, LDS_BYTES));
    init = true;
  }
  Params p;
  p.x = (const char*)d_in[0];
  p.w = (const char*)d_in[1];
  p.out = (char*)d_out;
  p.myws = (char*)d_ws_all[rank];
  p.rank = rank;
  p.pad = 0;
  for (int i = 0; i < NR; ++i) p.ws[i] = (char*)d_ws_all[i];
  hipLaunchKernelGGL(a2a_gemm, dim3(256), dim3(448), LDS_BYTES, stream, p);
  HIP_CHECK(hipGetLastError());
}
```
